```python
import numpy as np

kernel_name = "gpu_gemm_a2a_m4096_k4096_n4096_e4m3e5m2_dqrelu"


ABSMAX_RTOL = 0.05

COMPUTE_DTYPE = "float32"
EPILOGUE = "dequant_relu"
IN_QUANT = None
OUT_QUANT = None
IN_FP8 = "e4m3e5m2"
IN_DTYPES = {"x": "e4m3", "w_mat": "e5m2",
             "scale_x": "float32", "scale_w": "float32"}
WORLD_SIZE = 8

def _snap_fp8(x, mant_bits, max_abs, min_normal):
    a = np.abs(x.astype(np.float64))
    _m, _e = np.frexp(a)
    q = np.where(a >= min_normal, np.ldexp(1.0, _e - 1 - mant_bits),
                 min_normal * 2.0 ** -mant_bits)
    snapped = np.minimum(np.round(a / q) * q, max_abs)
    return (np.sign(x) * snapped).astype(np.float32)

def setup_inputs():

    x = _snap_fp8(np.random.randn(4096, 4096).astype(np.float32), 3, 448.0, 2.0 ** -6)
    w_mat = _snap_fp8(np.random.randn(4096, 4096).astype(np.float32), 2, 57344.0, 2.0 ** -14)
    scale_x = (0.02 * (1.0 + np.random.rand(1))).astype(np.float32)
    scale_w = (0.02 * (1.0 + np.random.rand(1))).astype(np.float32)
    return {"x": x, "w_mat": w_mat, "scale_x": scale_x, "scale_w": scale_w}

def reference(x, w_mat, scale_x, scale_w):
    acc = (x @ w_mat).astype(np.float32)
    y = (acc * (scale_x * scale_w)).astype(np.float32)
    return np.maximum(y, 0.0).astype(np.float32)
```

```cpp
#include <hip/hip_runtime.h>
#include <rccl/rccl.h>
#include <cstdio>
#include <utility>

#define HIP_CHECK(expr) do { hipError_t _e = (expr); if (_e != hipSuccess) { \
    fprintf(stderr, "HIP error %s at %s:%d\n", hipGetErrorString(_e), __FILE__, __LINE__); \
    return; } } while (0)

typedef int v2i __attribute__((ext_vector_type(2)));
typedef int v4i __attribute__((ext_vector_type(4)));
typedef int v8i __attribute__((ext_vector_type(8)));
typedef float v16f __attribute__((ext_vector_type(16)));
typedef float v4f __attribute__((ext_vector_type(4)));
#define LDS_AS __attribute__((address_space(3)))
#define CONST_AS __attribute__((address_space(4)))

constexpr int NR = 8;
constexpr int MS_ = 512;
constexpr int KD = 4096, ND = 4096;
constexpr int NSH = 512;
constexpr int TM = 64, TN = 128, BK = 128, NST = 5;
constexpr int A_BYTES = TM * BK, B_BYTES = BK * TN, ST_BYTES = A_BYTES + B_BYTES;
constexpr int NWAVE = 8, PPL = ST_BYTES / 1024 / NWAVE, PA = A_BYTES / 1024;
constexpr int NKT = KD / BK;
constexpr int LDS_BYTES = NST * ST_BYTES;
constexpr int GRID = 256;
constexpr unsigned long long DEADLINE_TICKS = 50000000ull;
constexpr int WS_CNT = 0;
constexpr int WS_TMO = GRID * 32;
constexpr int WS_FLAG = WS_TMO + 32;
constexpr int WS_STAMP = WS_FLAG + 256 * 32;

struct Params {
  const unsigned char* x; const unsigned char* w; const float* sx; const float* sw;
  float* out[NR]; unsigned* ws[NR];
  int rank;
};

template<int N, class F, int... I> __device__ __forceinline__ void static_for_impl(F&& f, std::integer_sequence<int, I...>) { (f(std::integral_constant<int, I>{}), ...); }
template<int N, class F> __device__ __forceinline__ void static_for(F&& f) { static_for_impl<N>(f, std::make_integer_sequence<int, N>{}); }
#define WAITV(n) asm volatile("s_waitcnt vmcnt(%0)" :: "n"(n) : "memory")

__global__ void __launch_bounds__(512, 1) fused_gemm_a2a(Params p) {
#if defined(__HIP_DEVICE_COMPILE__)
  extern __shared__ __attribute__((aligned(16))) unsigned char smem[];
  const int tid = threadIdx.x, lane = tid & 63;
  const int wid = __builtin_amdgcn_readfirstlane(tid >> 6);
  const int b = blockIdx.x, xcd = b & 7, idx = b >> 3;
  const int rank = p.rank;
  const int slice = xcd + 8 * (idx >> 3);
  const int m0 = (idx & 7) * TM, n0 = slice * TN;
  const unsigned dest = (unsigned)slice >> 2;
  v4i rx = {(int)(size_t)p.x, (int)(((size_t)p.x >> 32) & 0xffff), MS_ * KD, 0x00020000};
  v4i rw = {(int)(size_t)p.w, (int)(((size_t)p.w >> 32) & 0xffff), KD * ND, 0x00020000};
  int voff[PPL];
#pragma unroll
  for (int i = 0; i < PPL; ++i) {
    int pc = wid * PPL + i;
    if (pc < PA) {
      int r = pc * 8 + (lane >> 3);
      int ch = (lane & 7) ^ ((r >> 1) & 7);
      voff[i] = (m0 + r) * KD + ch * 16;
    } else {
      int kr = (pc - PA) * 8 + (lane >> 3);
      int ch = (lane & 7) ^ (((kr >> 1) & 3) << 1);
      voff[i] = kr * ND + n0 + ch * 16;
    }
  }
  const unsigned lds0 = (unsigned)(size_t)(LDS_AS unsigned char*)smem;
  auto issue = [&](int kt, int slotidx) {
    int soA = kt * BK, soB = kt * BK * ND;
    unsigned slot = lds0 + slotidx * ST_BYTES + wid * PPL * 1024;
#pragma unroll
    for (int i = 0; i < PPL; ++i) {
      int pc = wid * PPL + i;
      v4i rs = pc < PA ? rx : rw;
      int so = pc < PA ? soA : soB;
      asm volatile("s_mov_b32 m0, %0\n\ts_nop 0\n\tbuffer_load_dwordx4 %1, %2, %3 offen lds"
                   :: "s"(slot + i * 1024), "v"(voff[i]), "s"(rs), "s"(so) : "memory");
    }
  };
#pragma unroll 1
  for (int g = 0; g < NST; ++g) issue(g, g);
  unsigned* myws = p.ws[rank];
  float sxv = *(const CONST_AS float*)p.sx, swv = *(const CONST_AS float*)p.sw;
  unsigned call = *(const CONST_AS unsigned*)(myws + WS_CNT + b * 32);
  asm volatile("" : "+s"(sxv), "+s"(swv), "+s"(call));
  call += 1;
  const float scale = sxv * swv;

  const int wm = wid & 1, wn = wid >> 1;
  const int r = lane & 31, h = lane >> 5;
  const int arow = wm * 32 + r;
  const int aoff = arow * 128 + (((2 * h) ^ ((arow >> 1) & 7)) << 4);
  const int q = (lane & 15) >> 1;
  const int bkr = 32 * h + q;
  const int boff = A_BYTES + bkr * TN + (((wn * 2 + ((lane >> 4) & 1)) ^ (((bkr >> 1) & 3) << 1)) << 4) + 8 * (lane & 1);
  struct Frag { v8i a[2], b[2]; };
  auto ldfrag = [&](Frag& f, int slotidx) {
    const unsigned char* slot = smem + slotidx * ST_BYTES;
#pragma unroll
    for (int kk = 0; kk < 2; ++kk) {
      int o = aoff ^ (kk << 6);
      v4i lo = *(const v4i*)(slot + o), hi = *(const v4i*)(slot + (o ^ 16));
      f.a[kk] = __builtin_shufflevector(lo, hi, 0, 1, 2, 3, 4, 5, 6, 7);
      const unsigned char* bp = slot + boff + kk * 64 * TN;
      v2i b0 = __builtin_amdgcn_ds_read_tr8_b64_v2i32((LDS_AS v2i*)(bp));
      v2i b1 = __builtin_amdgcn_ds_read_tr8_b64_v2i32((LDS_AS v2i*)(bp + 8 * TN));
      v2i b2 = __builtin_amdgcn_ds_read_tr8_b64_v2i32((LDS_AS v2i*)(bp + 16 * TN));
      v2i b3 = __builtin_amdgcn_ds_read_tr8_b64_v2i32((LDS_AS v2i*)(bp + 24 * TN));
      v4i l = __builtin_shufflevector(b0, b1, 0, 1, 2, 3), hh = __builtin_shufflevector(b2, b3, 0, 1, 2, 3);
      f.b[kk] = __builtin_shufflevector(l, hh, 0, 1, 2, 3, 4, 5, 6, 7);
    }
  };
  v16f acc = {};
  Frag f0, f1;
  WAITV((NST - 1) * PPL);
  __builtin_amdgcn_s_barrier();
  asm volatile("" ::: "memory");
  ldfrag(f0, 0);
  int sl = 0;
  auto step = [&](auto remc, int g, Frag& cur, Frag& nxt) {
    constexpr int REM = decltype(remc)::value;
    constexpr bool DO_LD = REM >= 1, DO_ISSUE = REM >= NST;
    constexpr int INFL = (REM - 1 < NST - 2) ? (REM - 1 < 0 ? 0 : REM - 1) : NST - 2;
    if (DO_LD) {
      WAITV(INFL * PPL);
      __builtin_amdgcn_s_waitcnt(0xC07F);
      __builtin_amdgcn_sched_barrier(0);
      __builtin_amdgcn_s_barrier();
      asm volatile("" ::: "memory");
      __builtin_amdgcn_sched_barrier(0);
    }
    int sn = sl + 1 == NST ? 0 : sl + 1;
    if (DO_LD) ldfrag(nxt, sn);
    if (DO_ISSUE) issue(g + NST, sl);
    acc = __builtin_amdgcn_mfma_scale_f32_32x32x64_f8f6f4(cur.a[0], cur.b[0], acc, 0, 1, 0, 0, 0, 0);
    acc = __builtin_amdgcn_mfma_scale_f32_32x32x64_f8f6f4(cur.a[1], cur.b[1], acc, 0, 1, 0, 0, 0, 0);
    sl = sn;
  };
  constexpr int TAIL = (NST + 1) & ~1;
#pragma unroll 1
  for (int g = 0; g < NKT - TAIL; g += 2) { step(std::integral_constant<int, 100>{}, g, f0, f1); step(std::integral_constant<int, 100>{}, g + 1, f1, f0); }
  static_for<TAIL / 2>([&](auto ic) {
    constexpr int i = decltype(ic)::value;
    step(std::integral_constant<int, TAIL - 1 - 2 * i>{}, NKT - TAIL + 2 * i, f0, f1);
    step(std::integral_constant<int, TAIL - 2 - 2 * i>{}, NKT - TAIL + 2 * i + 1, f1, f0);
  });
  __builtin_amdgcn_s_barrier();
  float* eb = (float*)smem;
  constexpr int EP = TN + 4;
#pragma unroll
  for (int e = 0; e < 16; ++e) {
    int row = wm * 32 + (e & 3) + 8 * (e >> 2) + 4 * h;
    float v = acc[e] * scale;
    eb[row * EP + wn * 32 + r] = v > 0.f ? v : (v != v ? v : 0.f);
  }
  __syncthreads();
  if (dest < (unsigned)NR) {
    float* dst = p.out[dest] + (size_t)(rank * MS_ + m0) * NSH + (n0 & (NSH - 1));
#pragma unroll
    for (int i = 0; i < 4; ++i) {
      int row = wid * 8 + i * 2 + h;
      v4f v = *(const v4f*)(eb + row * EP + r * 4);
      float* d = dst + row * NSH + r * 4;
      asm volatile("global_store_dwordx4 %0, %1, off sc0 sc1\n\ts_nop 1" :: "v"(d), "v"(v) : "memory");
    }
  }
  asm volatile("s_waitcnt vmcnt(0)" ::: "memory");
  __syncthreads();
  if (tid == 0) {
    const int tile_in_dest = (slice & 3) * 8 + (idx & 7);
    if (dest != (unsigned)rank && dest < (unsigned)NR)
      __hip_atomic_store(p.ws[dest] + WS_FLAG + (rank * 32 + tile_in_dest) * 32, call, __ATOMIC_RELAXED, __HIP_MEMORY_SCOPE_SYSTEM);
    __hip_atomic_store(myws + WS_CNT + b * 32, call, __ATOMIC_RELAXED, __HIP_MEMORY_SCOPE_AGENT);
    if ((b >> 5) != rank) {
      unsigned* flag = myws + WS_FLAG + b * 32;
      unsigned* tmo = myws + WS_TMO;
      unsigned long long ts = __builtin_amdgcn_s_memrealtime();
      while (__hip_atomic_load(flag, __ATOMIC_RELAXED, __HIP_MEMORY_SCOPE_SYSTEM) < call) {
        if (__hip_atomic_load(tmo, __ATOMIC_RELAXED, __HIP_MEMORY_SCOPE_AGENT) == call || __builtin_amdgcn_s_memrealtime() - ts > DEADLINE_TICKS) {
          __hip_atomic_store(tmo, call, __ATOMIC_RELAXED, __HIP_MEMORY_SCOPE_AGENT);
          p.out[rank][(size_t)rank * MS_ * NSH] = __builtin_nanf("");
          break;
        }
        __builtin_amdgcn_s_sleep(2);
      }
    }
  }
#endif
}

extern "C" void kernel_launch(int rank, int nranks, ncclComm_t comm, hipStream_t stream,
                              void* const* d_in, const int* in_sizes, int n_in,
                              void* d_out, int out_size, const int* in_dtypes, int out_dtype,
                              void* const* d_out_all, void* const* d_ws_all, size_t ws_bytes) {
  if (nranks != NR) return;
  static bool init = false;
  if (!init) {
    HIP_CHECK(hipFuncSetAttribute((const void*)fused_gemm_a2a, hipFuncAttributeMaxDynamicSharedMemorySize, LDS_BYTES));
    init = true;
  }
  Params p;
  p.x = (const unsigned char*)d_in[0]; p.w = (const unsigned char*)d_in[1];
  p.sx = (const float*)d_in[2]; p.sw = (const float*)d_in[3];
  for (int i = 0; i < NR; ++i) { p.out[i] = (float*)d_out_all[i]; p.ws[i] = (unsigned*)d_ws_all[i]; }
  p.rank = rank;
  fused_gemm_a2a<<<GRID, 512, LDS_BYTES, stream>>>(p);
  HIP_CHECK(hipGetLastError());
}
```

```cpp
#include <hip/hip_runtime.h>
#include <rccl/rccl.h>
#include <cstdio>
#include <utility>

#define HIP_CHECK(expr) do { hipError_t _e = (expr); if (_e != hipSuccess) { \
    fprintf(stderr, "HIP error %s at %s:%d\n", hipGetErrorString(_e), __FILE__, __LINE__); \
    return; } } while (0)

typedef int v2i __attribute__((ext_vector_type(2)));
typedef int v4i __attribute__((ext_vector_type(4)));
typedef unsigned v4u __attribute__((ext_vector_type(4)));
typedef int v8i __attribute__((ext_vector_type(8)));
typedef float v16f __attribute__((ext_vector_type(16)));
typedef float v4f __attribute__((ext_vector_type(4)));
#define LDS_AS __attribute__((address_space(3)))
#define CONST_AS __attribute__((address_space(4)))

constexpr int NR = 8;
constexpr int MS_ = 512;
constexpr int KD = 4096, ND = 4096;
constexpr int NSH = 512;
constexpr int TM = 32, TN = 128, BK = 128;
#define NST 6
constexpr int A_BYTES = TM * BK, B_BYTES = BK * TN, ST_BYTES = A_BYTES + B_BYTES;
constexpr int NCW = 4, NLW = 4;
constexpr int PPL = ST_BYTES / 1024 / NLW, PA = A_BYTES / 1024;
constexpr int NKT = KD / BK;
constexpr int ROUNDS = 2;
constexpr int CHUNK_BYTES = 128 + 4096;
constexpr int RING_BYTES = NST * ST_BYTES;
constexpr int LDS_BYTES = RING_BYTES + NCW * CHUNK_BYTES;
static_assert(LDS_BYTES <= 160 * 1024, "lds");
static_assert(8 * CHUNK_BYTES <= RING_BYTES, "receive staging lives in the dead ring");
constexpr int GRID = 256;
constexpr unsigned long long DEADLINE_TICKS = 50000000ull;
constexpr size_t WS_CNT = 0;
constexpr size_t WS_TMO = GRID * 128;
constexpr size_t WS_STAMP = WS_TMO + 128;
constexpr size_t WS_SLOT = 256 * 1024;
constexpr size_t SLOT_SRC = 256 * (size_t)CHUNK_BYTES;
constexpr size_t WS_NEED = WS_SLOT + 2 * NR * SLOT_SRC;

struct Params {
  const unsigned char* x; const unsigned char* w; const float* sx; const float* sw;
  float* out; unsigned char* ws[NR];
  int rank;
};

template<int N, class F, int... I> __device__ __forceinline__ void static_for_impl(F&& f, std::integer_sequence<int, I...>) { (f(std::integral_constant<int, I>{}), ...); }
template<int N, class F> __device__ __forceinline__ void static_for(F&& f) { static_for_impl<N>(f, std::make_integer_sequence<int, N>{}); }
#define WAITV(n) asm volatile("s_waitcnt vmcnt(%0)" :: "n"(n) : "memory")
#define ST16(ptr, val) asm volatile("global_store_dwordx4 %0, %1, off sc0 sc1\n\ts_nop 1" :: "v"(ptr), "v"(val) : "memory")
#define STAMP(i)

__global__ void __launch_bounds__(512, 1) fused_gemm_a2a(Params p) {
#if defined(__HIP_DEVICE_COMPILE__)
  extern __shared__ __attribute__((aligned(16))) unsigned char smem[];
  const int tid = threadIdx.x, lane = tid & 63;
  const int wid_all = __builtin_amdgcn_readfirstlane(tid >> 6);
  const bool is_loader = wid_all >= NCW;
  const int wid = wid_all & 3;
  const int b = blockIdx.x, xcd = b & 7, idx = b >> 3;
  const int rank = p.rank;
  unsigned char* myws = p.ws[rank];
  const unsigned dest = (unsigned)(xcd + rank) & (NR - 1);
  const int tj = idx >> 3;
  const int n0 = dest * NSH + tj * TN;
  const int ti0 = (idx & 7) + 8 * (xcd & 1);
  const int rot = (((idx & 7) + (xcd >> 1)) & 3) * (NKT / 4);
  const unsigned lds0 = (unsigned)(size_t)(LDS_AS unsigned char*)smem;
  unsigned call;

  if (is_loader) {
    v4i rx = {(int)(size_t)p.x, (int)(((size_t)p.x >> 32) & 0xffff), MS_ * KD, 0x00020000};
    v4i rw = {(int)(size_t)p.w, (int)(((size_t)p.w >> 32) & 0xffff), KD * ND, 0x00020000};
    int voffA, voffB[4];
    { int r = wid * 8 + (lane >> 3); voffA = r * KD + (((lane & 7) ^ ((r >> 1) & 7)) << 4); }
#pragma unroll
    for (int i = 0; i < 4; ++i) {
      int kr = (wid * 4 + i) * 8 + (lane >> 3);
      voffB[i] = kr * ND + n0 + (((lane & 7) ^ (((kr >> 1) & 3) << 1)) << 4) - i * 1024;
    }
    auto issue = [&](int g, int slotidx) {
      int kt = (g + rot) & (NKT - 1);
      int ti = ti0 ^ ((g >> 5) << 3);
      int soA = ti * (TM * KD) + kt * BK, soB = kt * (BK * ND);
      unsigned slot = lds0 + slotidx * ST_BYTES;
      asm volatile("s_mov_b32 m0, %0\n\ts_nop 0\n\tbuffer_load_dwordx4 %1, %2, %3 offen lds"
                   :: "s"(slot + wid * 1024), "v"(voffA), "s"(rx), "s"(soA) : "memory");
      asm volatile("s_mov_b32 m0, %0\n\ts_nop 0\n\t"
                   "buffer_load_dwordx4 %1, %5, %6 offen lds\n\t"
                   "buffer_load_dwordx4 %2, %5, %6 offen offset:1024 lds\n\t"
                   "buffer_load_dwordx4 %3, %5, %6 offen offset:2048 lds\n\t"
                   "buffer_load_dwordx4 %4, %5, %6 offen offset:3072 lds"
                   :: "s"(slot + A_BYTES + wid * 4096), "v"(voffB[0]), "v"(voffB[1]), "v"(voffB[2]), "v"(voffB[3]), "s"(rw), "s"(soB) : "memory");
    };
#pragma unroll 1
    for (int g = 0; g < NST; ++g) issue(g, g);
    call = *(const CONST_AS unsigned*)(myws + WS_CNT + b * 128) + 1;
    WAITV((NST - 1) * PPL);
    __builtin_amdgcn_s_barrier();
    int sl = 0;
#pragma unroll 1
    for (int g = 0; g < ROUNDS * NKT - NST; ++g) {
      WAITV((NST - 2) * PPL);
      __builtin_amdgcn_s_barrier();
      issue(g + NST, sl);
      sl = sl + 1 == NST ? 0 : sl + 1;
    }
    static_for<NST - 1>([&](auto ic) {
      constexpr int i = decltype(ic)::value;
      WAITV((NST - 2 - i) * PPL);
      __builtin_amdgcn_s_barrier();
    });
    __builtin_amdgcn_s_barrier();
  } else {
    float sxv = *(const CONST_AS float*)p.sx, swv = *(const CONST_AS float*)p.sw;
    call = *(const CONST_AS unsigned*)(myws + WS_CNT + b * 128);
    asm volatile("" : "+s"(sxv), "+s"(swv), "+s"(call));
    call += 1;
    const float scale = sxv * swv;
    const int r = lane & 31, h = lane >> 5;
    const int aoff = r * 128 + (((2 * h) ^ ((r >> 1) & 7)) << 4);
    const int bkr = 32 * h + ((lane & 15) >> 1);
    const int boff = A_BYTES + bkr * TN + (((wid * 2 + ((lane >> 4) & 1)) ^ (((bkr >> 1) & 3) << 1)) << 4) + 8 * (lane & 1);
    struct Frag { v8i a[2], b[2]; };
    auto ldfrag = [&](Frag& f, int slotidx) {
      const unsigned char* slot = smem + slotidx * ST_BYTES;
#pragma unroll
      for (int kk = 0; kk < 2; ++kk) {
        int o = aoff ^ (kk << 6);
        v4i lo = *(const v4i*)(slot + o), hi = *(const v4i*)(slot + (o ^ 16));
        f.a[kk] = __builtin_shufflevector(lo, hi, 0, 1, 2, 3, 4, 5, 6, 7);
        const unsigned char* bp = slot + boff + kk * 64 * TN;
        v2i b0 = __builtin_amdgcn_ds_read_tr8_b64_v2i32((LDS_AS v2i*)(bp));
        v2i b1 = __builtin_amdgcn_ds_read_tr8_b64_v2i32((LDS_AS v2i*)(bp + 8 * TN));
        v2i b2 = __builtin_amdgcn_ds_read_tr8_b64_v2i32((LDS_AS v2i*)(bp + 16 * TN));
        v2i b3 = __builtin_amdgcn_ds_read_tr8_b64_v2i32((LDS_AS v2i*)(bp + 24 * TN));
        v4i l = __builtin_shufflevector(b0, b1, 0, 1, 2, 3), hh = __builtin_shufflevector(b2, b3, 0, 1, 2, 3);
        f.b[kk] = __builtin_shufflevector(l, hh, 0, 1, 2, 3, 4, 5, 6, 7);
      }
    };
    v16f acc;
    Frag f0, f1;
    int sl = 0;
    auto step = [&](Frag& cur, Frag& nxt) {
      __builtin_amdgcn_s_waitcnt(0xC07F);
      __builtin_amdgcn_sched_barrier(0);
      __builtin_amdgcn_s_barrier();
      asm volatile("" ::: "memory");
      __builtin_amdgcn_sched_barrier(0);
      sl = sl + 1 == NST ? 0 : sl + 1;
      ldfrag(nxt, sl);
      __builtin_amdgcn_sched_barrier(0);
      acc = __builtin_amdgcn_mfma_scale_f32_32x32x64_f8f6f4(cur.a[0], cur.b[0], acc, 0, 1, 0, 0, 0, 0);
      acc = __builtin_amdgcn_mfma_scale_f32_32x32x64_f8f6f4(cur.a[1], cur.b[1], acc, 0, 1, 0, 0, 0, 0);
      __builtin_amdgcn_sched_barrier(0);
    };
    unsigned* cb = (unsigned*)(smem + RING_BYTES + wid * CHUNK_BYTES);
    auto finish = [&](int round) {
      const int ti = ti0 ^ (round << 3);
      if (dest == (unsigned)rank) {
#pragma unroll
        for (int e = 0; e < 16; ++e) {
          float v = acc[e] * scale;
          v = v > 0.f ? v : (v != v ? v : 0.f);
          cb[((e & 3) + 8 * (e >> 2) + 4 * h) * 32 + r] = __float_as_uint(v);
        }
        float* dst = p.out + (size_t)(rank * MS_ + ti * TM + (lane >> 3)) * NSH + tj * TN + wid * 32 + (lane & 7) * 4;
#pragma unroll
        for (int i = 0; i < 4; ++i) {
          v4u v = *(const v4u*)(cb + i * 256 + lane * 4);
          ST16(dst + i * 8 * NSH, v);
        }
      } else {
        unsigned cnt = 32, hdr = 0;
        const unsigned cb_lds = lds0 + RING_BYTES + wid * CHUNK_BYTES;
#pragma unroll
        for (int e = 0; e < 16; ++e) {
          float v = acc[e] * scale;
          bool nz = !(v <= 0.f);
          unsigned long long m = __builtin_amdgcn_ballot_w64(nz);
          unsigned lo = max((unsigned)m, 1u), hi = max((unsigned)(m >> 32), 1u);
          unsigned u = nz ? __float_as_uint(v) : 0x80000000u;
          unsigned pos = __builtin_amdgcn_mbcnt_hi(hi, __builtin_amdgcn_mbcnt_lo(lo, cnt));
          unsigned long long m2 = ((unsigned long long)hi << 32) | lo;
          asm volatile("s_mov_b64 exec, %0\n\tds_write_b32 %1, %2\n\ts_mov_b64 exec, -1" :: "s"(m2), "v"(cb_lds + pos * 4), "v"(u) : "memory");
          asm volatile("v_writelane_b32 %0, %1, %3\n\tv_writelane_b32 %0, %2, %4" : "+v"(hdr) : "s"(lo), "s"(hi), "n"(2 * e), "n"(2 * e + 1));
          cnt += __builtin_popcount(lo) + __builtin_popcount(hi);
        }
        if (lane < 32) cb[lane] = hdr;
        if (dest < (unsigned)NR) {
          unsigned char* dst = p.ws[dest] + WS_SLOT + ((call & 1) * NR + rank) * SLOT_SRC + (size_t)(ti * 16 + tj * 4 + wid) * CHUNK_BYTES + lane * 16;
#pragma unroll
          for (int i = 0; i < 5; ++i) {
            if (i * 256u < cnt) {
              if (i * 256u + lane * 4u < cnt) {
                v4u v = *(const v4u*)(cb + i * 256 + lane * 4);
                ST16(dst + i * 1024, v);
              }
            }
          }
        }
      }
    };
    __builtin_amdgcn_s_barrier();
    asm volatile("" ::: "memory");
    ldfrag(f0, 0);
#pragma unroll 1
    for (int round = 0; round < ROUNDS; ++round) {
      acc = (v16f){};
#pragma unroll 1
      for (int g = 0; g < NKT; g += 2) { step(f0, f1); step(f1, f0); }
      STAMP(1 + 2 * round);
      finish(round);
      STAMP(2 + 2 * round);
    }
  }

  {
    const int u = wid_all;
    const int id = b * 4 + wid;
    const int si = id >> 7;
    if (si < 7) {
      const unsigned src = (unsigned)(rank + 1 + si) & (NR - 1);
      const int sx = (rank - (int)src) & (NR - 1);
      const int c7 = id & 127;
      const int ti = (c7 >> 4) + 8 * ((sx & 1) ^ (is_loader ? 0 : 1));
      const int cbk = c7 & 15;
      unsigned char* sp = myws + WS_SLOT + ((call & 1) * NR + src) * SLOT_SRC + (size_t)(ti * 16 + cbk) * CHUNK_BYTES;
      unsigned* rb = (unsigned*)(smem + u * CHUNK_BYTES);
      unsigned* tmo = (unsigned*)(myws + WS_TMO);
      unsigned long long ts = __builtin_amdgcn_s_memrealtime();
      bool dead = false;
      unsigned hdr;
      for (;;) {
        hdr = __hip_atomic_load((unsigned*)sp + (lane & 31), __ATOMIC_RELAXED, __HIP_MEMORY_SCOPE_SYSTEM);
        if (__builtin_amdgcn_ballot_w64(hdr == 0) == 0) break;
        if (__hip_atomic_load(tmo, __ATOMIC_RELAXED, __HIP_MEMORY_SCOPE_AGENT) == call || __builtin_amdgcn_s_memrealtime() - ts > DEADLINE_TICKS) { dead = true; break; }
        __builtin_amdgcn_s_sleep(4);
      }
      STAMP(5);
      unsigned pc = __builtin_popcount(hdr), inc = pc;
#pragma unroll
      for (int d = 1; d < 32; d <<= 1) { unsigned t = __shfl_up(inc, d, 32); if ((lane & 31) >= d) inc += t; }
      const unsigned cnt = 32 + __builtin_amdgcn_readlane(inc, 31);
      const unsigned pre = 32 + inc - pc;
      v4u o[4];
      while (!dead) {
        v4u d[5];
#pragma unroll
        for (int i = 0; i < 5; ++i) {
          d[i] = (v4u){0, 0, 0, 0};
          if (i * 256u < cnt && i * 256u + lane * 4u < cnt)
            asm volatile("global_load_dwordx4 %0, %1, off sc0 sc1" : "=v"(d[i]) : "v"(sp + i * 1024 + lane * 16) : "memory");
        }
        asm volatile("s_waitcnt vmcnt(0)" : "+v"(d[0]), "+v"(d[1]), "+v"(d[2]), "+v"(d[3]), "+v"(d[4]) :: "memory");
#pragma unroll
        for (int i = 0; i < 5; ++i) if (i * 256u < cnt && (i < 4 || lane < 8)) *(v4u*)(rb + i * 256 + lane * 4) = d[i];
        bool bad = false;
#pragma unroll
        for (int i = 0; i < 4; ++i) {
          const int rl = lane >> 3;
          const int hw = 2 * ((rl & 3) + 4 * i) + (rl >> 2);
          unsigned m = __shfl(hdr, hw, 64), base = __shfl(pre, hw, 64);
          const int c0 = (lane & 7) * 4;
          base += __builtin_popcount(m & ((1u << c0) - 1u));
          unsigned nib = m >> c0;
#pragma unroll
          for (int j = 0; j < 4; ++j) {
            unsigned v = rb[base];
            bool on = (nib >> j) & 1;
            bad |= on && v == 0;
            o[i][j] = on ? (v == 0x80000000u ? 0u : v) : 0u;
            base += on;
          }
        }
        if (__builtin_amdgcn_ballot_w64(bad) == 0) break;
        if (__hip_atomic_load(tmo, __ATOMIC_RELAXED, __HIP_MEMORY_SCOPE_AGENT) == call || __builtin_amdgcn_s_memrealtime() - ts > DEADLINE_TICKS) dead = true;
      }
      if (dead) {
        __hip_atomic_store(tmo, call, __ATOMIC_RELAXED, __HIP_MEMORY_SCOPE_AGENT);
#pragma unroll
        for (int i = 0; i < 4; ++i) o[i] = (v4u){0x7fc00000u, 0x7fc00000u, 0x7fc00000u, 0x7fc00000u};
      }
      float* dst = p.out + (size_t)(src * MS_ + ti * TM + (lane >> 3)) * NSH + cbk * 32 + (lane & 7) * 4;
#pragma unroll
      for (int i = 0; i < 4; ++i) ST16(dst + i * 8 * NSH, o[i]);
      v4u z = {0, 0, 0, 0};
#pragma unroll
      for (int i = 0; i < 5; ++i)
        if (i * 256u < cnt && i * 256u + lane * 4u < cnt) ST16(sp + i * 1024 + lane * 16, z);
      STAMP(6);
    }
  }
  if (tid == 0) __hip_atomic_store((unsigned*)(myws + WS_CNT + b * 128), call, __ATOMIC_RELAXED, __HIP_MEMORY_SCOPE_AGENT);
#endif
}

extern "C" void kernel_launch(int rank, int nranks, ncclComm_t comm, hipStream_t stream,
                              void* const* d_in, const int* in_sizes, int n_in,
                              void* d_out, int out_size, const int* in_dtypes, int out_dtype,
                              void* const* d_out_all, void* const* d_ws_all, size_t ws_bytes) {
  if (nranks != NR || ws_bytes < WS_NEED || (unsigned)rank >= (unsigned)NR) return;
  static bool init = false;
  if (!init) {
    HIP_CHECK(hipFuncSetAttribute((const void*)fused_gemm_a2a, hipFuncAttributeMaxDynamicSharedMemorySize, LDS_BYTES));
    init = true;
  }
  Params p;
  p.x = (const unsigned char*)d_in[0]; p.w = (const unsigned char*)d_in[1];
  p.sx = (const float*)d_in[2]; p.sw = (const float*)d_in[3];
  p.out = (float*)d_out;
  for (int i = 0; i < NR; ++i) p.ws[i] = (unsigned char*)d_ws_all[i];
  p.rank = rank;
  fused_gemm_a2a<<<GRID, 512, LDS_BYTES, stream>>>(p);
  HIP_CHECK(hipGetLastError());
}
```

```cpp
#include <hip/hip_runtime.h>
#include <rccl/rccl.h>
#include <cstdio>
#include <utility>

#define HIP_CHECK(expr) do { hipError_t _e = (expr); if (_e != hipSuccess) { \
    fprintf(stderr, "HIP error %s at %s:%d\n", hipGetErrorString(_e), __FILE__, __LINE__); \
    return; } } while (0)

typedef int v2i __attribute__((ext_vector_type(2)));
typedef int v4i __attribute__((ext_vector_type(4)));
typedef unsigned v4u __attribute__((ext_vector_type(4)));
typedef unsigned v2u __attribute__((ext_vector_type(2)));
typedef int v8i __attribute__((ext_vector_type(8)));
typedef float v16f __attribute__((ext_vector_type(16)));
#define LDS_AS __attribute__((address_space(3)))
#define CONST_AS __attribute__((address_space(4)))
#define GLB_AS __attribute__((address_space(1)))
typedef GLB_AS unsigned gu32;

constexpr int NR = 8;
constexpr int MS_ = 512;
constexpr int KD = 4096, ND = 4096;
constexpr int NSH = 512;
constexpr int TM = 64, TN = 128, BK = 128;
#define NST 5
#define XWIN 2
#define PSLEEP 4
#define FLAGW 1
#define APLAIN 0
#define LMAP 1
#define YP0 512
#define YT_LO 0
#define YT_HI 1000
#define YTAIL 128
#define SDELAY 0
#define XSLEEP 8
#define XHOLD 0
#define NOX 0
#define FSLEEP 0
#define YP1 512
#define YWIN 0
constexpr int YROWS = 384, XROWS = MS_ - YROWS;
#define XEARLY 6
#define XFL_STRIDE 32
#define XE3 4
constexpr int XFMIN = XE3;
static_assert(XE3 >= 4 && XEARLY >= XE3 && XWIN <= 3, "");
constexpr int A_BYTES = TM * BK, B_BYTES = BK * TN, ST_BYTES = A_BYTES + B_BYTES;
constexpr int PPL = 6;
constexpr int NKT = KD / BK;
constexpr int CHUNK_BYTES = 128 + 4096;
constexpr int RBUF_BYTES = CHUNK_BYTES + 256;
constexpr int CBUF_BYTES = CHUNK_BYTES + 16 + 128;
constexpr int LDS_DEAD = NST * ST_BYTES;
constexpr int LDS_BYTES = NST * ST_BYTES + 32;
static_assert(LDS_BYTES <= 160 * 1024, "lds");
static_assert(8 * CBUF_BYTES + 4 * RBUF_BYTES <= LDS_BYTES, "epilogue buffers live in the dead ring");
constexpr int GRID = 256;
constexpr unsigned long long DEADLINE_TICKS = 50000000ull;
constexpr size_t WS_CNT = 0;
constexpr size_t WS_TMO = GRID * 128;
constexpr size_t WS_XFLAG = WS_TMO + 128;
constexpr size_t WS_STAMP = 128 * 1024;
constexpr size_t WS_XS = 512 * 1024;
constexpr size_t XS_SRC = (size_t)XROWS * KD;
constexpr size_t WS_SLOT = WS_XS + 2 * NR * XS_SRC;
constexpr size_t SLOT_SRC = (YROWS / 32) * 16 * (size_t)CHUNK_BYTES;
constexpr size_t WS_NEED = WS_SLOT + 2 * NR * SLOT_SRC;

struct Params {
  const unsigned char* x; const unsigned char* w; unsigned char* myws; int rank; int pad;
  const float* sx; const float* sw; float* out;
  unsigned char* ws[NR];
};

struct XSched { int wait, flag; };
constexpr int xflag_at(int i) { int done = i - XWIN; return (done > 0 && (done & 3) == 0 && (done / 4 == XE3 || done / 4 == XEARLY)) ? done / 4 : 0; }
constexpr XSched xsched(int i) {
  int pos = 8, ldpos[32] = {}, stpos[32] = {};
  for (int j = 0; j < 8; ++j) ldpos[j] = j;
  for (int j = 0; j < i; ++j) {
    if (xflag_at(j)) ++pos;
    stpos[j] = pos++;
    if ((j & 3) == 0 && j / 4 + 2 < 8) for (int q = 0; q < 4; ++q) ldpos[(j / 4 + 2) * 4 + q] = pos++;
  }
  int need = ldpos[i];
  if (i - XWIN - 1 >= 0 && stpos[i - XWIN - 1] > need) need = stpos[i - XWIN - 1];
  return XSched{pos - 1 - need, xflag_at(i)};
}
template<int N, class F, int... I> __device__ __forceinline__ void static_for_impl(F&& f, std::integer_sequence<int, I...>) { (f(std::integral_constant<int, I>{}), ...); }
template<int N, class F> __device__ __forceinline__ void static_for(F&& f) { static_for_impl<N>(f, std::make_integer_sequence<int, N>{}); }
__device__ __forceinline__ unsigned wave_scan(unsigned v) {
#if defined(__HIP_DEVICE_COMPILE__)
  v += __builtin_amdgcn_update_dpp(0u, v, 0x111, 0xf, 0xf, true);
  v += __builtin_amdgcn_update_dpp(0u, v, 0x112, 0xf, 0xf, true);
  v += __builtin_amdgcn_update_dpp(0u, v, 0x114, 0xf, 0xf, true);
  v += __builtin_amdgcn_update_dpp(0u, v, 0x118, 0xf, 0xf, true);
  v += __builtin_amdgcn_update_dpp(0u, v, 0x142, 0xa, 0xf, false);
  v += __builtin_amdgcn_update_dpp(0u, v, 0x143, 0xc, 0xf, false);
#endif
  return v;
}
#define WAITV(n) asm volatile("s_waitcnt vmcnt(%0)" :: "n"(n) : "memory")
#define ST16(ptr, val) asm volatile("global_store_dwordx4 %0, %1, off sc0 sc1\n\ts_nop 1" :: "v"(ptr), "v"(val) : "memory")
__device__ __forceinline__ void st16(void* ptr, v4u val) {
#if defined(__HIP_DEVICE_COMPILE__)
  ST16(ptr, val);
#endif
}
template<int KB> __device__ __forceinline__ void x_ldblk(v4u (&d)[32], const unsigned char* sp) {
#if defined(__HIP_DEVICE_COMPILE__)
  asm volatile("global_load_dwordx4 %0, %4, off offset:%6\n\tglobal_load_dwordx4 %1, %5, off offset:%6\n\t"
               "global_load_dwordx4 %2, %7, off offset:%6\n\tglobal_load_dwordx4 %3, %8, off offset:%6"
               : "=&v"(d[KB * 4]), "=&v"(d[KB * 4 + 1]), "=&v"(d[KB * 4 + 2]), "=&v"(d[KB * 4 + 3])
               : "v"(sp), "v"(sp + 2 * KD), "n"(KB * 512), "v"(sp + 4 * KD), "v"(sp + 6 * KD) : "memory");
#endif
}
template<int N> __device__ __forceinline__ void x_wait(v4u& v) {
#if defined(__HIP_DEVICE_COMPILE__)
  asm volatile("s_waitcnt vmcnt(%1)" : "+v"(v) : "n"(N) : "memory");
#endif
}
#define STAMP(i)

__global__ void __launch_bounds__(512, 1) fused_gemm_a2a(Params p) {
#if defined(__HIP_DEVICE_COMPILE__)
  extern __shared__ __attribute__((aligned(16))) unsigned char smem[];
  const int tid = threadIdx.x, lane = tid & 63;
  const int wid_all = __builtin_amdgcn_readfirstlane(tid >> 6);
  const bool is_loader = wid_all >= 4;
  const int wid = wid_all & 3;
  const int b = blockIdx.x, xcd = b & 7, idx = b >> 3;
  int rank = p.rank;
  unsigned char* myws = p.myws;
  const unsigned char* px = p.x; const unsigned char* pw_ = p.w;
  asm volatile("" : "+s"(rank), "+s"(myws), "+s"(px), "+s"(pw_));
  STAMP(0);
  const unsigned peer = (unsigned)(xcd + rank) & (NR - 1);
  const bool roleS = xcd == 0, roleL = !roleS && idx < 8, roleY = !roleS && !roleL;
  int ti, tj;
  unsigned srcL = 0;
  if (roleS) { ti = idx & 7; tj = idx >> 3; }
  else if (roleL) { ti = idx & 1; tj = idx >> 1; srcL = (unsigned)(rank - xcd) & (NR - 1); }
  else { tj = (idx - 8) / 6; ti = (idx - 8) - tj * 6; }
  const int n0 = (roleY ? peer : rank) * NSH + tj * TN;
#define ROTM 4
#if ROTM == 1
  const int rot = roleL ? 0 : ((ti + tj) & 3) * (NKT / 4);
#elif ROTM == 2
  const int rot = roleL ? 0 : (roleS ? 4 * ti : (ti * 16 + 1) / 3) + 8 * tj;
#elif ROTM == 3
  const int rot = roleL ? 0 : (roleS ? 4 * ti : (ti * 16 + 1) / 3) + 8 * tj + xcd;
#elif ROTM == 4
  const int rot = roleL ? 0 : (roleS ? 4 * ti : (ti * 16 + 1) / 3) + 8 * xcd;
#else
  const int rot = roleL ? 0 : ((ti + xcd) & 3) * (NKT / 4);
#endif
  const unsigned lds0 = (unsigned)(size_t)(LDS_AS unsigned char*)smem;
  gu32* tmo = (gu32*)(myws + WS_TMO);
  const CONST_AS unsigned* cntp = (const CONST_AS unsigned*)(myws + WS_CNT + b * 128);
  unsigned call = 0;
  bool dead = false;

  auto give_up = [&](unsigned long long t0) -> bool {
    const unsigned long long dt = __builtin_amdgcn_s_memrealtime() - t0;
    if (dt < 2000) return false;
    if (dt > DEADLINE_TICKS) return true;
    __builtin_amdgcn_s_sleep(64);
    unsigned tv = 0;
    if (lane == 0) tv = __hip_atomic_load(tmo, __ATOMIC_RELAXED, __HIP_MEMORY_SCOPE_AGENT);
    return __builtin_amdgcn_readfirstlane(tv) == call;
  };
  gu32* xfl = (gu32*)(myws + WS_XFLAG) + ((srcL * 16 + ti * 8 + (is_loader ? wid * 2 + (lane & 1) : (lane & 7))) * 4 + tj) * 32;
  const int xearly = tj == 3 ? XE3 : XEARLY;
  auto wait_xflags = [&](unsigned nblk, int slot) {
    unsigned long long t0 = __builtin_amdgcn_s_memrealtime();
    const unsigned want = 8 * (call - 1) + nblk;
    unsigned fv = want, dd = 0;
    for (;;) {
      asm volatile("s_mov_b64 exec, %2\n\tglobal_load_dword %0, %1, off sc0 sc1\n\ts_mov_b64 exec, -1\n\ts_waitcnt vmcnt(0)"
                   : "+v"(fv) : "v"(xfl), "s"(slot < 4 ? 0x3ull : 0xffull) : "memory");
      if (__builtin_amdgcn_ballot_w64(fv < want) == 0) break;
      if (give_up(t0)) {
        __hip_atomic_store(tmo, call, __ATOMIC_RELAXED, __HIP_MEMORY_SCOPE_AGENT);
        WAITV(0);
        dd = 1;
        break;
      }
      __builtin_amdgcn_s_sleep(XSLEEP);
    }
    *(LDS_AS unsigned*)(size_t)(lds0 + LDS_DEAD + slot * 4) = dd;
    __builtin_amdgcn_s_waitcnt(0xC07F);
  };
  if (__builtin_expect(roleL, 0)) {
    if (is_loader) {
      call = *cntp + 1;
      wait_xflags(xearly, wid);
    } else if (idx < 4) {
      const int xo = (idx * 32 + wid * 8 + (lane >> 5)) * KD + (lane & 31) * 16;
      const unsigned char* sp = px + (size_t)YROWS * KD + xo;
      v4u d[32];
      x_ldblk<0>(d, sp); x_ldblk<1>(d, sp);
      call = *cntp + 1;
      if (peer < (unsigned)NR) {
        unsigned char* pw = p.ws[peer];
        unsigned char* dp = pw + WS_XS + ((call & 1) * NR + rank) * XS_SRC + xo;
        gu32* pf = (gu32*)(pw + WS_XFLAG) + ((rank * 16 + idx * 4 + wid) * 4 + (lane & 3)) * 32;
        const unsigned fbase = 8 * (call - 1);
        static_for<32>([&](auto ic) {
          constexpr int i = decltype(ic)::value;
          constexpr XSched xs = xsched(i);
          x_wait<xs.wait>(d[i]);
          if (xs.flag) {
            if (lane < 4) __hip_atomic_store(pf, fbase + xs.flag, __ATOMIC_RELAXED, __HIP_MEMORY_SCOPE_SYSTEM);
            if (xs.flag == XE3) STAMP(5);
          }
          if (!NOX || lane == 0) st16(dp + (i >> 2) * 512 + (i & 3) * 2 * KD, d[i]);
          if (i == 0) STAMP(6);
          if constexpr ((i & 3) == 0 && i / 4 + 2 < 8) x_ldblk<i / 4 + 2>(d, sp);
        });
        WAITV(0);
        if (lane < 4) __hip_atomic_store(pf, fbase + 8, __ATOMIC_RELAXED, __HIP_MEMORY_SCOPE_SYSTEM);
      }
    }
    STAMP(1);
  }

  if (__builtin_expect(is_loader, 1)) {
    const unsigned char* abase = roleL ? myws + WS_XS + ((call & 1) * NR + srcL) * XS_SRC : px;
    v4i rx = {(int)(size_t)abase, (int)(((size_t)abase >> 32) & 0xffff), (roleL ? XROWS : MS_) * KD, 0x00020000};
    v4i rw = {(int)(size_t)pw_, (int)(((size_t)pw_ >> 32) & 0xffff), KD * ND, 0x00020000};
    int voffA[2], voffB[4];
#pragma unroll
    for (int i = 0; i < 2; ++i) { int r = (wid * 2 + i) * 8 + (lane >> 3); voffA[i] = (ti * TM + r) * KD + (((lane & 7) ^ ((r >> 1) & 7)) << 4) - i * 1024; }
#pragma unroll
    for (int i = 0; i < 4; ++i) {
      int kr = (wid * 4 + i) * 8 + (lane >> 3);
      voffB[i] = kr * ND + n0 + (((lane & 7) ^ (((kr >> 1) & 3) << 1)) << 4) - i * 1024;
    }
    auto issue = [&](int g, int slotidx) {
      int kt = (g + rot) & (NKT - 1);
      if (roleL && g < 16) kt = g ^ (ti << 3);
      int soA = kt * BK, soB = kt * (BK * ND);
      unsigned slot = lds0 + slotidx * ST_BYTES;
      if (roleL && !APLAIN)
        asm volatile("s_mov_b32 m0, %0\n\ts_nop 0\n\tbuffer_load_dwordx4 %1, %3, %4 offen sc0 sc1 lds\n\tbuffer_load_dwordx4 %2, %3, %4 offen offset:1024 sc0 sc1 lds"
                     :: "s"(slot + wid * 2048), "v"(voffA[0]), "v"(voffA[1]), "s"(rx), "s"(soA) : "memory");
      else
        asm volatile("s_mov_b32 m0, %0\n\ts_nop 0\n\tbuffer_load_dwordx4 %1, %3, %4 offen lds\n\tbuffer_load_dwordx4 %2, %3, %4 offen offset:1024 lds"
                     :: "s"(slot + wid * 2048), "v"(voffA[0]), "v"(voffA[1]), "s"(rx), "s"(soA) : "memory");
      asm volatile("s_mov_b32 m0, %0\n\ts_nop 0\n\t"
                   "buffer_load_dwordx4 %1, %5, %6 offen lds\n\t"
                   "buffer_load_dwordx4 %2, %5, %6 offen offset:1024 lds\n\t"
                   "buffer_load_dwordx4 %3, %5, %6 offen offset:2048 lds\n\t"
                   "buffer_load_dwordx4 %4, %5, %6 offen offset:3072 lds"
                   :: "s"(slot + A_BYTES + wid * 4096), "v"(voffB[0]), "v"(voffB[1]), "v"(voffB[2]), "v"(voffB[3]), "s"(rw), "s"(soB) : "memory");
    };
#pragma unroll 1
    for (int g = 0; g < NST; ++g) issue(g, g);
    call = *cntp + 1;
    WAITV((NST - 1) * PPL);
    __builtin_amdgcn_s_barrier();
    int sl = 0;
#pragma unroll 1
    for (int g = 0; g < NKT - NST; ++g) {
      WAITV((NST - 2) * PPL);
      __builtin_amdgcn_s_barrier();
      issue(g + NST, sl);
      sl = sl + 1 == NST ? 0 : sl + 1;
    }
    static_for<NST - 1>([&](auto ic) {
      constexpr int i = decltype(ic)::value;
      WAITV((NST - 2 - i) * PPL);
      __builtin_amdgcn_s_barrier();
    });
    __builtin_amdgcn_s_barrier();
    STAMP(3);

    if (roleY) {
      const int id = ((xcd - 1) * 24 + (idx - 8)) * 4 + wid;
      const int si = id / 96, c96 = id - si * 96;
      const unsigned src = (unsigned)(rank + 1 + si) & (NR - 1);
      const int cbk = c96 & 15;
      unsigned* rb = (unsigned*)(smem + 8 * CBUF_BYTES + wid * RBUF_BYTES);
      unsigned* tab = rb + CHUNK_BYTES / 4;
      const unsigned rb_lds = lds0 + 8 * CBUF_BYTES + wid * RBUF_BYTES;
      unsigned long long t0 = __builtin_amdgcn_s_memrealtime();
#pragma unroll 1
      for (int blk = 0; blk < 2; ++blk) {
        const int rb32 = (c96 >> 4) * 2 + blk;
        unsigned char* sp0 = myws + WS_SLOT + ((call & 1) * NR + src) * SLOT_SRC + (size_t)(rb32 * 16 + cbk) * CHUNK_BYTES;
        v4u hd = {~0u, ~0u, ~0u, ~0u};
        for (;;) {
          asm volatile("s_mov_b64 exec, 0xff\n\tglobal_load_dwordx4 %0, %1, off sc0 sc1\n\ts_mov_b64 exec, -1\n\ts_waitcnt vmcnt(0)" : "+v"(hd) : "v"(sp0 + lane * 16) : "memory");
          const unsigned lo = min(min(hd[0] & 0xffffu, hd[1] & 0xffffu), min(hd[2] & 0xffffu, hd[3] & 0xffffu));
          const unsigned hi = min(min(hd[0], hd[1]), min(hd[2], hd[3]));
          if (__builtin_amdgcn_ballot_w64(lo == 0 || hi < 0x10000u) == 0) break;
          if (dead || give_up(t0)) { dead = true; break; }
          __builtin_amdgcn_s_sleep(PSLEEP);
        }
        if (lane < 8) *(v4u*)(rb + lane * 4) = hd;
        const unsigned msk = ((const unsigned short*)rb)[lane];
        if (blk == 1) STAMP(4);
        const unsigned pc = __builtin_popcount(msk);
        const unsigned inc = wave_scan(pc);
        const unsigned nby = (128 + 4 * __builtin_amdgcn_readlane(inc, 63) + 15) & ~15u;
        const v4i rs = {(int)(size_t)sp0, (int)(((size_t)sp0 >> 32) & 0xffff), (int)nby, 0x00020000};
        const bool big = nby > 128 + 2048;
        v4u d[4];
        unsigned of[4];
#pragma unroll
        for (int i = 0; i < 4; ++i) of[i] = 128 + i * 1024u + lane * 16u;
        d[2] = d[3] = (v4u){0, 0, 0, 0};
        auto fetch = [&]() {
          asm volatile("s_nop 4\n\tbuffer_load_dwordx4 %0, %2, %4, 0 offen sc0 sc1\n\tbuffer_load_dwordx4 %1, %3, %4, 0 offen sc0 sc1"
                       : "=&v"(d[0]), "=&v"(d[1]) : "v"(of[0]), "v"(of[1]), "s"(rs) : "memory");
          if (big)
            asm volatile("buffer_load_dwordx4 %0, %2, %4, 0 offen sc0 sc1\n\tbuffer_load_dwordx4 %1, %3, %4, 0 offen sc0 sc1"
                         : "+v"(d[2]), "+v"(d[3]) : "v"(of[2]), "v"(of[3]), "s"(rs) : "memory");
        };
        fetch();
        tab[lane] = ((32 + inc - pc) << 16) | msk;
        unsigned ga[4][4], gm[4][4];
#pragma unroll
        for (int i = 0; i < 4; ++i) {
          const unsigned t = tab[i * 8 + (lane >> 3) + 32 * (lane & 1)];
          const unsigned sh = (lane & 6) * 2;
          unsigned a = rb_lds + 4 * ((t >> 16) + __builtin_popcount(t & ((1u << sh) - 1u)));
          const unsigned nib = t >> sh;
          ga[i][0] = a;
          asm("v_bfe_i32 %0, %7, 0, 1\n\tv_bfe_i32 %1, %7, 1, 1\n\tv_bfe_i32 %2, %7, 2, 1\n\tv_bfe_i32 %3, %7, 3, 1\n\t"
              "v_mad_i32_i24 %4, %0, -4, %8\n\tv_mad_i32_i24 %5, %1, -4, %4\n\tv_mad_i32_i24 %6, %2, -4, %5"
              : "=&v"(gm[i][0]), "=&v"(gm[i][1]), "=&v"(gm[i][2]), "=&v"(gm[i][3]), "=&v"(ga[i][1]), "=&v"(ga[i][2]), "=&v"(ga[i][3]) : "v"(nib), "v"(a));
        }
        for (;;) {
          asm volatile("s_waitcnt vmcnt(0)" : "+v"(d[0]), "+v"(d[1]), "+v"(d[2]), "+v"(d[3]) :: "memory");
          bool miss = false;
#pragma unroll
          for (int i = 0; i < 4; ++i) miss |= of[i] < nby && min(min(d[i][0], d[i][1]), min(d[i][2], d[i][3])) == 0;
          if (__builtin_amdgcn_ballot_w64(miss) == 0 || dead) break;
          if (give_up(t0)) { dead = true; break; }
          __builtin_amdgcn_s_sleep(FSLEEP);
          fetch();
        }
        if (blk == 1) STAMP(5);
#pragma unroll
        for (int i = 0; i < 2; ++i) *(v4u*)((unsigned char*)rb + of[i]) = d[i];
        if (big) { *(v4u*)((unsigned char*)rb + of[2]) = d[2]; *(v4u*)((unsigned char*)rb + of[3]) = d[3]; }
        float* dst = p.out + (size_t)(src * MS_ + rb32 * 32 + (lane >> 3)) * NSH + cbk * 32 + (lane & 7) * 4;
        const unsigned nanv = dead ? 0x7fc00000u : 0u;
        v4u o[4];
#pragma unroll
        for (int i = 0; i < 4; ++i)
#pragma unroll
          for (int j = 0; j < 4; ++j) o[i][j] = *(const LDS_AS unsigned*)(size_t)ga[i][j];
#pragma unroll
        for (int i = 0; i < 4; ++i) {
#pragma unroll
          for (int j = 0; j < 4; ++j) o[i][j] = (o[i][j] & gm[i][j]) | nanv;
          ST16(dst + i * 8 * NSH, o[i]);
        }
        const v4u zz = {0, 0, 0, 0};
        const unsigned lo16 = lane * 16;
        asm volatile("buffer_store_dwordx4 %0, %1, %2, 0 offen sc0 sc1\n\tbuffer_store_dwordx4 %0, %1, %2, 0 offen offset:1024 sc0 sc1\n\t"
                     "buffer_store_dwordx4 %0, %1, %2, 0 offen offset:2048 sc0 sc1\n\ts_nop 1" :: "v"(zz), "v"(lo16), "s"(rs) : "memory");
        if (nby > 3072)
          asm volatile("buffer_store_dwordx4 %0, %1, %2, 0 offen offset:3072 sc0 sc1\n\tbuffer_store_dwordx4 %0, %3, %2, 0 offen sc0 sc1\n\ts_nop 1"
                       :: "v"(zz), "v"(lo16), "s"(rs), "v"(lo16 + 4096) : "memory");
      }
      if (dead) __hip_atomic_store(tmo, call, __ATOMIC_RELAXED, __HIP_MEMORY_SCOPE_AGENT);
      STAMP(6);
    }
  } else {
    float sxv = *(const CONST_AS float*)p.sx, swv = *(const CONST_AS float*)p.sw;
    call = *cntp + 1;
    unsigned char* pws = p.ws[peer & (NR - 1)];
    float* pout = p.out;
    asm volatile("" : "+s"(sxv), "+s"(swv), "+s"(call), "+s"(pws), "+s"(pout));
    const float scale = sxv * swv;
    const int r = lane & 31, h = lane >> 5;
    const int aoff = r * 128 + (((2 * h) ^ ((r >> 1) & 7)) << 4);
    const int bkr = 32 * h + ((lane & 15) >> 1);
    const int boff = A_BYTES + bkr * TN + (((wid * 2 + ((lane >> 4) & 1)) ^ (((bkr >> 1) & 3) << 1)) << 4) + 8 * (lane & 1);
    struct Frag { v8i a[2][2], b[2]; };
    auto ldfrag = [&](Frag& f, int slotidx) {
      const unsigned char* slot = smem + slotidx * ST_BYTES;
#pragma unroll
      for (int kk = 0; kk < 2; ++kk) {
        const unsigned char* bp = slot + boff + kk * 64 * TN;
        v2i b0 = __builtin_amdgcn_ds_read_tr8_b64_v2i32((LDS_AS v2i*)(bp));
        v2i b1 = __builtin_amdgcn_ds_read_tr8_b64_v2i32((LDS_AS v2i*)(bp + 8 * TN));
        v2i b2 = __builtin_amdgcn_ds_read_tr8_b64_v2i32((LDS_AS v2i*)(bp + 16 * TN));
        v2i b3 = __builtin_amdgcn_ds_read_tr8_b64_v2i32((LDS_AS v2i*)(bp + 24 * TN));
        v4i l = __builtin_shufflevector(b0, b1, 0, 1, 2, 3), hh = __builtin_shufflevector(b2, b3, 0, 1, 2, 3);
        f.b[kk] = __builtin_shufflevector(l, hh, 0, 1, 2, 3, 4, 5, 6, 7);
        int o = aoff ^ (kk << 6);
#pragma unroll
        for (int i = 0; i < 2; ++i) {
          v4i lo = *(const v4i*)(slot + o + i * 4096), hi = *(const v4i*)(slot + (o ^ 16) + i * 4096);
          f.a[kk][i] = __builtin_shufflevector(lo, hi, 0, 1, 2, 3, 4, 5, 6, 7);
        }
      }
    };
    v16f acc[2] = {};
    Frag f0, f1;
    int sl = 0;
    auto step = [&](Frag& cur, Frag& nxt) {
      __builtin_amdgcn_s_waitcnt(0xC07F);
      __builtin_amdgcn_sched_barrier(0);
      __builtin_amdgcn_s_barrier();
      asm volatile("" ::: "memory");
      __builtin_amdgcn_sched_barrier(0);
      sl = sl + 1 == NST ? 0 : sl + 1;
      ldfrag(nxt, sl);
#pragma unroll
      for (int kk = 0; kk < 2; ++kk)
#pragma unroll
        for (int i = 0; i < 2; ++i)
          acc[i] = __builtin_amdgcn_mfma_scale_f32_32x32x64_f8f6f4(cur.b[kk], cur.a[kk][i], acc[i], 1, 0, 0, 0, 0, 0);
#pragma unroll
      for (int i = 0; i < 4; ++i) { __builtin_amdgcn_sched_group_barrier(0x8, 1, 0); __builtin_amdgcn_sched_group_barrier(0x100, 4, 0); }
      __builtin_amdgcn_sched_barrier(0);
    };
    __builtin_amdgcn_s_barrier();
    asm volatile("" ::: "memory");
    STAMP(2);
    ldfrag(f0, 0);
#pragma unroll 1
    for (int g = 0; g < NKT; g += 2) {
      if (g == ((xearly * 4 - NST) & ~1) && roleL && wid == 0) wait_xflags(8, 4);
      step(f0, f1); step(f1, f0);
    }
    STAMP(3);
    if (!roleY) {
      const bool bad = roleL && __builtin_amdgcn_ballot_w64(*(const LDS_AS unsigned*)(size_t)(lds0 + LDS_DEAD + min(lane, 4) * 4) != 0) != 0;
      const int orow = roleL ? srcL * MS_ + YROWS + ti * TM : rank * MS_ + ti * TM;
      unsigned char* cb = smem + wid * 2 * CBUF_BYTES;
      const float sc2 = bad ? __builtin_nanf("") : scale;
#pragma unroll
      for (int i = 0; i < 2; ++i)
#pragma unroll
        for (int q = 0; q < 4; ++q) {
          v4u o;
#pragma unroll
          for (int j = 0; j < 4; ++j) { float v = acc[i][q * 4 + j] * sc2; o[j] = !(v <= 0.f) ? __float_as_uint(v) : 0u; }
          *(v4u*)(cb + (i * 32 + r) * 128 + (((q * 2 + h) ^ (r & 7)) << 4)) = o;
        }
      float* dst = pout + (size_t)(orow + (lane >> 3)) * NSH + tj * TN + wid * 32 + (lane & 7) * 4;
      v4u o[8];
#pragma unroll
      for (int i = 0; i < 8; ++i) o[i] = *(const v4u*)(cb + i * 1024 + (lane >> 3) * 128 + (((lane & 7) ^ (lane >> 3)) << 4));
#pragma unroll
      for (int i = 0; i < 8; ++i) ST16(dst + i * 8 * NSH, o[i]);
      STAMP(4);
    } else if (peer < (unsigned)NR) {
      unsigned char* dbase = pws + WS_SLOT + ((call & 1) * NR + rank) * SLOT_SRC + (size_t)(ti * 32 + tj * 4 + wid) * CHUNK_BYTES;
      unsigned nbys[2];
      auto send = [&](int blk, unsigned off, unsigned end, bool last) {
        const unsigned char* db = dbase + blk * 16 * CHUNK_BYTES;
        const v4i rs = {(int)(size_t)db, (int)(((size_t)db >> 32) & 0xffff), (int)end, 0x00020000};
        const unsigned o = (last && lane >= 56) ? (lane - 56) * 16 : off + lane * 16;
        v4u v = *(const v4u*)(smem + (wid * 2 + blk) * CBUF_BYTES + o);
        asm volatile("s_nop 4\n\tbuffer_store_dwordx4 %0, %1, %2, 0 offen sc0 sc1\n\ts_nop 1" :: "v"(v), "v"(o), "s"(rs) : "memory");
      };
      unsigned off = 128;
      auto piece = [&](int blk, unsigned nby, unsigned sz) {
        const bool last = nby - off <= min(sz + YTAIL, 896u);
        const unsigned end = last ? nby : off + min(sz, nby - off - 16);
        send(blk, off, end, last);
        off = end;
      };
      unsigned long long tsend;
#pragma unroll
      for (int blk = 0; blk < 2; ++blk) {
        unsigned* cb = (unsigned*)(smem + (wid * 2 + blk) * CBUF_BYTES);
        const unsigned cb_lds = lds0 + (wid * 2 + blk) * CBUF_BYTES;
        float u[16];
        unsigned msk = 0;
#pragma unroll
        for (int e = 0; e < 16; ++e) u[e] = acc[blk][e] * scale;
#define CA(n) "v_cmp_nge_f32 vcc, 0, %" #n "\n\tv_addc_co_u32 %0, vcc, %0, %0, vcc\n\t"
        asm volatile(CA(16) CA(15) CA(14) CA(13) CA(12) CA(11) CA(10) CA(9) CA(8) CA(7) CA(6) CA(5) CA(4) CA(3) CA(2) CA(1)
            : "+v"(msk) : "v"(u[0]), "v"(u[1]), "v"(u[2]), "v"(u[3]), "v"(u[4]), "v"(u[5]), "v"(u[6]), "v"(u[7]),
              "v"(u[8]), "v"(u[9]), "v"(u[10]), "v"(u[11]), "v"(u[12]), "v"(u[13]), "v"(u[14]), "v"(u[15]) : "vcc");
#undef CA
        if (msk == 0) u[0] = -0.0f;
        msk = max(msk, 1u);
        const unsigned pc = __builtin_popcount(msk);
        const unsigned inc = wave_scan(pc);
        *(LDS_AS unsigned short*)(size_t)(cb_lds + lane * 2) = (unsigned short)msk;
        unsigned a = cb_lds + 4 * (32 + inc - pc), t0_, t1_;
        const unsigned trash = cb_lds + CHUNK_BYTES + 16 + r * 4;
#define PKE(e, n) "v_bfe_i32 %1, %3, " #e ", 1\n\tv_bfi_b32 %2, %1, %0, %4\n\tv_mad_i32_i24 %0, %1, -4, %0\n\tds_write_b32 %2, %" #n "\n\t"
        asm volatile(PKE(0, 5) PKE(1, 6) PKE(2, 7) PKE(3, 8) PKE(4, 9) PKE(5, 10) PKE(6, 11) PKE(7, 12) PKE(8, 13) PKE(9, 14) PKE(10, 15) PKE(11, 16) PKE(12, 17) PKE(13, 18) PKE(14, 19) PKE(15, 20)
            : "+v"(a), "=&v"(t0_), "=&v"(t1_) : "v"(msk), "v"(trash), "v"(u[0]), "v"(u[1]), "v"(u[2]), "v"(u[3]), "v"(u[4]), "v"(u[5]), "v"(u[6]), "v"(u[7]),
              "v"(u[8]), "v"(u[9]), "v"(u[10]), "v"(u[11]), "v"(u[12]), "v"(u[13]), "v"(u[14]), "v"(u[15]) : "memory");
#undef PKE
        const unsigned cnt = 32 + __builtin_amdgcn_readlane(inc, 63);
        if (lane < 3) cb[cnt + lane] = 1u;
        nbys[blk] = (cnt * 4 + 15) & ~15u;
        if (blk == 0) {
          piece(0, nbys[0], YP0);
          tsend = __builtin_amdgcn_s_memrealtime();
          STAMP(4);
          __builtin_amdgcn_sched_barrier(0);
        }
      }
#pragma unroll 1
      for (int blk = 0; blk < 2; ++blk) {
        const unsigned nby = blk ? nbys[1] : nbys[0];
        while (off < nby) {
          WAITV(0);
          const unsigned long long now = __builtin_amdgcn_s_memrealtime();
          const unsigned rt = (unsigned)(now - tsend);
          piece(blk, nby, rt < YT_LO ? 1024u : (rt < YT_HI ? (unsigned)YP1 : 256u));
          tsend = now;
        }
        off = 128;
      }
      STAMP(5);
    }
  }
  if (tid == 0) __hip_atomic_store((gu32*)(myws + WS_CNT + b * 128), call, __ATOMIC_RELAXED, __HIP_MEMORY_SCOPE_AGENT);
#endif
}

extern "C" void kernel_launch(int rank, int nranks, ncclComm_t comm, hipStream_t stream,
                              void* const* d_in, const int* in_sizes, int n_in,
                              void* d_out, int out_size, const int* in_dtypes, int out_dtype,
                              void* const* d_out_all, void* const* d_ws_all, size_t ws_bytes) {
  if (nranks != NR || ws_bytes < WS_NEED || (unsigned)rank >= (unsigned)NR) return;
  static bool init = false;
  if (!init) {
    HIP_CHECK(hipFuncSetAttribute((const void*)fused_gemm_a2a, hipFuncAttributeMaxDynamicSharedMemorySize, LDS_BYTES));
    init = true;
  }
  Params p;
  p.x = (const unsigned char*)d_in[0]; p.w = (const unsigned char*)d_in[1];
  p.sx = (const float*)d_in[2]; p.sw = (const float*)d_in[3];
  p.out = (float*)d_out;
  for (int i = 0; i < NR; ++i) p.ws[i] = (unsigned char*)d_ws_all[i];
  p.myws = p.ws[rank];
  p.rank = rank; p.pad = 0;
  fused_gemm_a2a<<<GRID, 512, LDS_BYTES, stream>>>(p);
  HIP_CHECK(hipGetLastError());
}
```

```cpp
#include <hip/hip_runtime.h>
#include <rccl/rccl.h>
#include <cstdio>
#include <utility>

#define HIP_CHECK(expr) do { hipError_t _e = (expr); if (_e != hipSuccess) { \
    fprintf(stderr, "HIP error %s at %s:%d\n", hipGetErrorString(_e), __FILE__, __LINE__); \
    return; } } while (0)

typedef int v2i __attribute__((ext_vector_type(2)));
typedef int v4i __attribute__((ext_vector_type(4)));
typedef unsigned v4u __attribute__((ext_vector_type(4)));
typedef int v8i __attribute__((ext_vector_type(8)));
typedef float v16f __attribute__((ext_vector_type(16)));
#define LDS_AS __attribute__((address_space(3)))
#define CONST_AS __attribute__((address_space(4)))
#define GLB_AS __attribute__((address_space(1)))
typedef GLB_AS unsigned gu32;

constexpr int NR = 8;
constexpr int MS_ = 512;
constexpr int KD = 4096, ND = 4096;
constexpr int NSH = 512;
constexpr int TM = 64, TN = 128, BK = 128;
constexpr int NST = 5;
constexpr int YROWS = 384, XROWS = MS_ - YROWS;
constexpr int XWIN = 4;
constexpr int XTAILW = 8;
constexpr int XEARLY = 4;
constexpr unsigned YPIECE = 768, YTAIL = 128;
constexpr int A_BYTES = TM * BK, B_BYTES = BK * TN, ST_BYTES = A_BYTES + B_BYTES;
constexpr int PPL = 6;
constexpr int NKT = KD / BK;
constexpr int CHUNK_BYTES = 128 + 4096;
constexpr int RBUF_BYTES = CHUNK_BYTES + 256;
constexpr int CBUF_BYTES = CHUNK_BYTES + 16 + 128;
constexpr int LDS_DEAD = NST * ST_BYTES;
constexpr int LDS_XFL = LDS_DEAD + 32;
constexpr int LDS_CALL = LDS_XFL + 32;
constexpr int LDS_BYTES = LDS_CALL + 16;
static_assert(LDS_BYTES <= 160 * 1024, "lds");
static_assert(8 * CBUF_BYTES + 4 * RBUF_BYTES + 1024 <= LDS_DEAD, "epilogue buffers live in the dead ring");
static_assert(XEARLY >= 4 && XEARLY < 8 && YPIECE + YTAIL <= 1024, "");
constexpr int GRID = 256;
constexpr unsigned long long DEADLINE_TICKS = 50000000ull;
constexpr size_t WS_CNT = 0;
constexpr size_t WS_TMO = GRID * 128;
constexpr size_t WS_XFLAG = WS_TMO + 128;
constexpr size_t WS_STAMP = 128 * 1024;
constexpr size_t WS_XS = 512 * 1024;
constexpr size_t XS_SRC = (size_t)XROWS * KD;
constexpr size_t WS_SLOT = WS_XS + 2 * NR * XS_SRC;
constexpr size_t SLOT_SRC = (YROWS / 32) * 16 * (size_t)CHUNK_BYTES;
constexpr size_t WS_NEED = WS_SLOT + 2 * NR * SLOT_SRC;
static_assert(WS_XFLAG + NR * 16 * 4 * 128 <= WS_STAMP, "");

struct Params {
  const unsigned char* x; const unsigned char* w; unsigned char* myws; int rank; int pad;
  const float* sx; const float* sw; float* out;
  unsigned char* ws[NR];
};

struct XSched { int wait, flag; };
constexpr int xflag_at(int i) { return i - XWIN == 4 * XEARLY ? XEARLY : 0; }
constexpr XSched xsched(int i) {
  int pos = 8, ldpos[32] = {}, stpos[32] = {};
  for (int j = 0; j < 8; ++j) ldpos[j] = j;
  for (int j = 0; j < i; ++j) {
    if (xflag_at(j)) ++pos;
    stpos[j] = pos++;
    if ((j & 3) == 0 && j / 4 + 2 < 8) for (int q = 0; q < 4; ++q) ldpos[(j / 4 + 2) * 4 + q] = pos++;
  }
  int need = ldpos[i];
  if (i - XWIN - 1 >= 0 && i < 32 - XTAILW && stpos[i - XWIN - 1] > need) need = stpos[i - XWIN - 1];
  return XSched{pos - 1 - need, xflag_at(i)};
}
static_assert(4 * XEARLY + XWIN < 32 - XTAILW, "the early progress word is published behind a throttled wait");

template<int N, class F, int... I> __device__ __forceinline__ void static_for_impl(F&& f, std::integer_sequence<int, I...>) { (f(std::integral_constant<int, I>{}), ...); }
template<int N, class F> __device__ __forceinline__ void static_for(F&& f) { static_for_impl<N>(f, std::make_integer_sequence<int, N>{}); }

__device__ __forceinline__ unsigned wave_scan(unsigned v) {
#if defined(__HIP_DEVICE_COMPILE__)
  v += __builtin_amdgcn_update_dpp(0u, v, 0x111, 0xf, 0xf, true);
  v += __builtin_amdgcn_update_dpp(0u, v, 0x112, 0xf, 0xf, true);
  v += __builtin_amdgcn_update_dpp(0u, v, 0x114, 0xf, 0xf, true);
  v += __builtin_amdgcn_update_dpp(0u, v, 0x118, 0xf, 0xf, true);
  v += __builtin_amdgcn_update_dpp(0u, v, 0x142, 0xa, 0xf, false);
  v += __builtin_amdgcn_update_dpp(0u, v, 0x143, 0xc, 0xf, false);
#endif
  return v;
}
#define WAITV(n) asm volatile("s_waitcnt vmcnt(%0)" :: "n"(n) : "memory")
__device__ __forceinline__ void st16(void* ptr, v4u val) {
#if defined(__HIP_DEVICE_COMPILE__)
  asm volatile("global_store_dwordx4 %0, %1, off sc0 sc1\n\ts_nop 1" :: "v"(ptr), "v"(val) : "memory");
#endif
}
template<int KB> __device__ __forceinline__ void x_ldblk(v4u (&d)[32], const unsigned char* sp) {
#if defined(__HIP_DEVICE_COMPILE__)
  asm volatile("global_load_dwordx4 %0, %4, off offset:%6\n\tglobal_load_dwordx4 %1, %5, off offset:%6\n\t"
               "global_load_dwordx4 %2, %7, off offset:%6\n\tglobal_load_dwordx4 %3, %8, off offset:%6"
               : "=&v"(d[KB * 4]), "=&v"(d[KB * 4 + 1]), "=&v"(d[KB * 4 + 2]), "=&v"(d[KB * 4 + 3])
               : "v"(sp), "v"(sp + 2 * KD), "n"(KB * 512), "v"(sp + 4 * KD), "v"(sp + 6 * KD) : "memory");
#endif
}
template<int N, class T> __device__ __forceinline__ void vm_wait(T& v) {
#if defined(__HIP_DEVICE_COMPILE__)
  asm volatile("s_waitcnt vmcnt(%1)" : "+v"(v) : "n"(N) : "memory");
#endif
}
__device__ __forceinline__ v4i make_rsrc(const void* base, unsigned bytes) {
  return (v4i){(int)(size_t)base, (int)(((size_t)base >> 32) & 0xffff), (int)bytes, 0x00020000};
}
#define STAMP(i)

__global__ void __launch_bounds__(512, 1) fused_gemm_a2a(Params p) {
#if defined(__HIP_DEVICE_COMPILE__)
  extern __shared__ __attribute__((aligned(16))) unsigned char smem[];
  const int tid = threadIdx.x, lane = tid & 63;
  const int wid_all = __builtin_amdgcn_readfirstlane(tid >> 6);
  const bool is_loader = wid_all >= 4;
  const int wid = wid_all & 3;
  const int b = blockIdx.x, xcd = b & 7, idx = b >> 3;
  int rank = p.rank;
  unsigned char* myws = p.myws;
  const unsigned char* px = p.x; const unsigned char* pw_ = p.w;
  asm volatile("" : "+s"(rank), "+s"(myws), "+s"(px), "+s"(pw_));
  STAMP(0);
  gu32* cntp = (gu32*)(myws + WS_CNT + b * 128);
  const unsigned peer = (unsigned)(xcd + rank) & (NR - 1);
  const unsigned srcL = (unsigned)(rank - xcd) & (NR - 1);
  const bool roleS = xcd == 0, roleL = !roleS && idx < 8, roleY = !roleS && !roleL;
  int ti, tj;
  if (roleS) { ti = idx & 7; tj = idx >> 3; }
  else if (roleL) { ti = idx & 1; tj = idx >> 1; }
  else { tj = (idx - 8) / 6; ti = (idx - 8) - tj * 6; }
  const int n0 = (roleY ? peer : rank) * NSH + tj * TN;
  const int rot = roleL ? 0 : (roleS ? 4 * ti : (ti * 16 + 1) / 3) + 8 * xcd;
  const unsigned lds0 = (unsigned)(size_t)(LDS_AS unsigned char*)smem;
  gu32* tmo = (gu32*)(myws + WS_TMO);
  unsigned call = 0;

  auto give_up = [&](unsigned long long t0) -> bool {
    const unsigned long long dt = __builtin_amdgcn_s_memrealtime() - t0;
    if (dt < 2000) return false;
    if (dt > DEADLINE_TICKS) return true;
    __builtin_amdgcn_s_sleep(64);
    unsigned tv = 0;
    if (lane == 0) tv = __hip_atomic_load(tmo, __ATOMIC_RELAXED, __HIP_MEMORY_SCOPE_AGENT);
    return __builtin_amdgcn_readfirstlane(tv) == call;
  };
  gu32* xfl = (gu32*)(myws + WS_XFLAG) + ((srcL * 16 + ti * 8 + wid * 2 + (lane & 1)) * 4 + tj) * 32;
  auto wait_xflags = [&](unsigned nblk, int slot) {
    const unsigned long long t0 = __builtin_amdgcn_s_memrealtime();
    const unsigned want = 8 * (call - 1) + nblk;
    unsigned fv = want, dd = 0;
    for (;;) {
      asm volatile("s_mov_b64 exec, 3\n\tglobal_load_dword %0, %1, off sc0 sc1\n\ts_mov_b64 exec, -1\n\ts_waitcnt vmcnt(0)"
                   : "+v"(fv) : "v"(xfl) : "memory");
      if (__builtin_amdgcn_ballot_w64(fv < want) == 0) break;
      if (give_up(t0)) {
        __hip_atomic_store(tmo, call, __ATOMIC_RELAXED, __HIP_MEMORY_SCOPE_AGENT);
        WAITV(0);
        dd = 1;
        break;
      }
      __builtin_amdgcn_s_sleep(8);
    }
    *(LDS_AS unsigned*)(size_t)(lds0 + LDS_DEAD + slot * 4) = dd;
    __builtin_amdgcn_s_waitcnt(0xC07F);
  };

  if (__builtin_expect(roleL, 0)) {
    if (is_loader) {
      call = __builtin_amdgcn_readfirstlane(__hip_atomic_load(cntp, __ATOMIC_RELAXED, __HIP_MEMORY_SCOPE_AGENT)) + 1;
      *(LDS_AS unsigned*)(size_t)(lds0 + LDS_DEAD + 16 + wid * 4) = 0;
      wait_xflags(XEARLY, wid);
    } else if (idx < 4) {
      const int xo = (idx * 32 + wid * 8 + (lane >> 5)) * KD + (lane & 31) * 16;
      const unsigned char* sp = px + (size_t)YROWS * KD + xo;
      v4u d[32];
      unsigned cntv;
      asm volatile("global_load_dword %0, %1, off sc1" : "=v"(cntv) : "v"(cntp) : "memory");
      x_ldblk<0>(d, sp); x_ldblk<1>(d, sp);
      unsigned char* pw = p.ws[peer & (NR - 1)];
      asm volatile("s_waitcnt vmcnt(8)\n\tv_readfirstlane_b32 %0, %1" : "=s"(call) : "v"(cntv) : "memory");
      call += 1;
      unsigned char* dp = pw + WS_XS + ((call & 1) * NR + rank) * XS_SRC + xo;
      gu32* pf = (gu32*)(pw + WS_XFLAG) + ((rank * 16 + idx * 4 + wid) * 4 + (lane & 3)) * 32;
      const unsigned fbase = 8 * (call - 1);
      static_for<32>([&](auto ic) {
        constexpr int i = decltype(ic)::value;
        constexpr XSched xs = xsched(i);
        vm_wait<xs.wait>(d[i]);
        if (xs.flag) {
          if (lane < 4) __hip_atomic_store(pf, fbase + xs.flag, __ATOMIC_RELAXED, __HIP_MEMORY_SCOPE_SYSTEM);
          STAMP(5);
        }
        st16(dp + (i >> 2) * 512 + (i & 3) * 2 * KD, d[i]);
        if (i == 0) STAMP(6);
        if constexpr ((i & 3) == 0 && i / 4 + 2 < 8) x_ldblk<i / 4 + 2>(d, sp);
      });
      WAITV(0);
      if (lane < 4) __hip_atomic_store(pf, fbase + 8, __ATOMIC_RELAXED, __HIP_MEMORY_SCOPE_SYSTEM);
    }
    STAMP(1);
  }

  if (__builtin_expect(is_loader, 1)) {
    const unsigned char* abase = roleL ? myws + WS_XS + ((call & 1) * NR + srcL) * XS_SRC : px;
    const v4i rx = make_rsrc(abase, (roleL ? XROWS : MS_) * KD);
    const v4i rw = make_rsrc(pw_, KD * ND);
    int voffA[2], voffB[4];
#pragma unroll
    for (int i = 0; i < 2; ++i) { int r = (wid * 2 + i) * 8 + (lane >> 3); voffA[i] = (ti * TM + r) * KD + (((lane & 7) ^ ((r >> 1) & 7)) << 4) - i * 1024; }
#pragma unroll
    for (int i = 0; i < 4; ++i) {
      int kr = (wid * 4 + i) * 8 + (lane >> 3);
      voffB[i] = kr * ND + n0 + (((lane & 7) ^ (((kr >> 1) & 3) << 1)) << 4) - i * 1024;
    }
    auto issue = [&](const bool L, int kt, unsigned slot) __attribute__((always_inline)) {
      const int soA = kt * BK, soB = kt * (BK * ND);
      if (L)
        asm volatile("s_mov_b32 m0, %0\n\ts_nop 0\n\tbuffer_load_dwordx4 %1, %3, %4 offen sc0 sc1 lds\n\tbuffer_load_dwordx4 %2, %3, %4 offen offset:1024 sc0 sc1 lds"
                     :: "s"(slot + wid * 2048), "v"(voffA[0]), "v"(voffA[1]), "s"(rx), "s"(soA) : "memory");
      else
        asm volatile("s_mov_b32 m0, %0\n\ts_nop 0\n\tbuffer_load_dwordx4 %1, %3, %4 offen lds\n\tbuffer_load_dwordx4 %2, %3, %4 offen offset:1024 lds"
                     :: "s"(slot + wid * 2048), "v"(voffA[0]), "v"(voffA[1]), "s"(rx), "s"(soA) : "memory");
      asm volatile("s_mov_b32 m0, %0\n\ts_nop 0\n\t"
                   "buffer_load_dwordx4 %1, %5, %6 offen lds\n\t"
                   "buffer_load_dwordx4 %2, %5, %6 offen offset:1024 lds\n\t"
                   "buffer_load_dwordx4 %3, %5, %6 offen offset:2048 lds\n\t"
                   "buffer_load_dwordx4 %4, %5, %6 offen offset:3072 lds"
                   :: "s"(slot + A_BYTES + wid * 4096), "v"(voffB[0]), "v"(voffB[1]), "v"(voffB[2]), "v"(voffB[3]), "s"(rw), "s"(soB) : "memory");
    };
    auto ring = [&](const bool L) __attribute__((always_inline)) {
      auto kstage = [&](int g) { return L ? (g < 16 ? g ^ (ti << 3) : g) : ((g + rot) & (NKT - 1)); };
#pragma unroll 1
      for (int g = 0; g < NST; ++g) issue(L, kstage(g), lds0 + g * ST_BYTES);
      WAITV((NST - 1) * PPL);
      __builtin_amdgcn_s_barrier();
      unsigned slot = lds0;
#pragma unroll 1
      for (int g = NST; g < NKT; ++g) {
        int kt = kstage(g);
        asm volatile("s_waitcnt vmcnt(%1)" : "+s"(kt) : "n"((NST - 2) * PPL) : "memory");
        __builtin_amdgcn_s_barrier();
        if (L && g == XEARLY * 4) {
          const unsigned fv = *(const LDS_AS unsigned*)(size_t)(lds0 + LDS_XFL + wid * 8 + (lane & 1) * 4);
          if (__builtin_amdgcn_ballot_w64(fv < 8 * call) != 0) wait_xflags(8, 4 + wid);
        }
        issue(L, kt, slot);
        if (L && g == XEARLY * 4 - (NST - 1))
          asm volatile("s_mov_b32 m0, %0\n\ts_mov_b64 exec, 3\n\tglobal_load_lds_dword %1, off sc0 sc1\n\ts_mov_b64 exec, -1"
                       :: "s"(lds0 + LDS_XFL + wid * 8), "v"(xfl) : "memory");
        slot = slot + ST_BYTES == lds0 + NST * ST_BYTES ? lds0 : slot + ST_BYTES;
      }
    };
    if (__builtin_expect(roleL, 0)) ring(true); else ring(false);
    static_for<NST - 1>([&](auto ic) {
      constexpr int i = decltype(ic)::value;
      WAITV((NST - 2 - i) * PPL);
      __builtin_amdgcn_s_barrier();
    });
    __builtin_amdgcn_s_barrier();
    asm volatile("" ::: "memory");
    STAMP(3);

    if (roleY) {
      call = __builtin_amdgcn_readfirstlane(*(const LDS_AS unsigned*)(size_t)(lds0 + LDS_CALL));
      const int id = ((xcd - 1) * 24 + (idx - 8)) * 4 + wid;
      const int si = id / 96, c96 = id - si * 96;
      const unsigned src = (unsigned)(rank + 1 + si) & (NR - 1);
      const int cbk = c96 & 15;
      unsigned* rb = (unsigned*)(smem + 8 * CBUF_BYTES + wid * RBUF_BYTES);
      unsigned* tab = rb + CHUNK_BYTES / 4;
      const unsigned rb_lds = lds0 + 8 * CBUF_BYTES + wid * RBUF_BYTES;
      const unsigned long long t0 = __builtin_amdgcn_s_memrealtime();
      bool dead = false;
#pragma unroll 1
      for (int blk = 0; blk < 2; ++blk) {
        const int rb32 = (c96 >> 4) * 2 + blk;
        unsigned char* sp0 = myws + WS_SLOT + ((call & 1) * NR + src) * SLOT_SRC + (size_t)(rb32 * 16 + cbk) * CHUNK_BYTES;
        v4u hd = {~0u, ~0u, ~0u, ~0u};
        for (;;) {
          asm volatile("s_mov_b64 exec, 0xff\n\tglobal_load_dwordx4 %0, %1, off sc0 sc1\n\ts_mov_b64 exec, -1\n\ts_waitcnt vmcnt(0)" : "+v"(hd) : "v"(sp0 + lane * 16) : "memory");
          const unsigned lo = min(min(hd[0] & 0xffffu, hd[1] & 0xffffu), min(hd[2] & 0xffffu, hd[3] & 0xffffu));
          const unsigned hi = min(min(hd[0], hd[1]), min(hd[2], hd[3]));
          if (__builtin_amdgcn_ballot_w64(lo == 0 || hi < 0x10000u) == 0) break;
          if (dead || give_up(t0)) { dead = true; break; }
          __builtin_amdgcn_s_sleep(4);
        }
        if (lane < 8) *(v4u*)(rb + lane * 4) = hd;
        const unsigned msk = ((const unsigned short*)rb)[lane];
        if (blk == 1) STAMP(4);
        const unsigned pc = __builtin_popcount(msk);
        const unsigned inc = wave_scan(pc);
        const unsigned nby = (128 + 4 * __builtin_amdgcn_readlane(inc, 63) + 15) & ~15u;
        const v4i rs = make_rsrc(sp0, nby);
        v4u d[4];
        unsigned of[4];
#pragma unroll
        for (int i = 0; i < 4; ++i) { of[i] = 128 + i * 1024u + lane * 16u; d[i] = (v4u){0, 0, 0, 0}; }
        unsigned pend = (nby > 128 + 3072) ? 15u : (nby > 128 + 2048) ? 7u : (nby > 128 + 1024) ? 3u : 1u;
        auto fetch = [&]() {
#pragma unroll
          for (int i = 0; i < 4; ++i)
            if (pend >> i & 1) asm volatile("s_nop 4\n\tbuffer_load_dwordx4 %0, %1, %2, 0 offen sc0 sc1" : "+v"(d[i]) : "v"(of[i]), "s"(rs) : "memory");
        };
        fetch();
        tab[lane] = ((32 + inc - pc) << 16) | msk;
        unsigned ga[4][4], gm[4][4];
#pragma unroll
        for (int i = 0; i < 4; ++i) {
          const unsigned t = tab[i * 8 + (lane >> 3) + 32 * (lane & 1)];
          const unsigned sh = (lane & 6) * 2;
          const unsigned a = rb_lds + 4 * ((t >> 16) + __builtin_popcount(t & ((1u << sh) - 1u)));
          const unsigned nib = t >> sh;
          ga[i][0] = a;
          asm("v_bfe_i32 %0, %7, 0, 1\n\tv_bfe_i32 %1, %7, 1, 1\n\tv_bfe_i32 %2, %7, 2, 1\n\tv_bfe_i32 %3, %7, 3, 1\n\t"
              "v_mad_i32_i24 %4, %0, -4, %8\n\tv_mad_i32_i24 %5, %1, -4, %4\n\tv_mad_i32_i24 %6, %2, -4, %5"
              : "=&v"(gm[i][0]), "=&v"(gm[i][1]), "=&v"(gm[i][2]), "=&v"(gm[i][3]), "=&v"(ga[i][1]), "=&v"(ga[i][2]), "=&v"(ga[i][3]) : "v"(nib), "v"(a));
        }
        for (;;) {
          asm volatile("s_waitcnt vmcnt(0)" : "+v"(d[0]), "+v"(d[1]), "+v"(d[2]), "+v"(d[3]) :: "memory");
#pragma unroll
          for (int i = 0; i < 4; ++i)
            if (pend >> i & 1) {
              const bool miss = of[i] < nby && min(min(d[i][0], d[i][1]), min(d[i][2], d[i][3])) == 0;
              if (__builtin_amdgcn_ballot_w64(miss) == 0) { pend &= ~(1u << i); *(v4u*)((unsigned char*)rb + of[i]) = d[i]; }
            }
          if (pend == 0 || dead) break;
          if (give_up(t0)) { dead = true; break; }
          fetch();
        }
        if (blk == 1) STAMP(5);
        float* dst = p.out + (size_t)(src * MS_ + rb32 * 32 + (lane >> 3)) * NSH + cbk * 32 + (lane & 7) * 4;
        const unsigned nanv = dead ? 0x7fc00000u : 0u;
        v4u o[4];
#pragma unroll
        for (int i = 0; i < 4; ++i)
#pragma unroll
          for (int j = 0; j < 4; ++j) o[i][j] = *(const LDS_AS unsigned*)(size_t)ga[i][j];
#pragma unroll
        for (int i = 0; i < 4; ++i) {
#pragma unroll
          for (int j = 0; j < 4; ++j) o[i][j] = (o[i][j] & gm[i][j]) | nanv;
          st16(dst + i * 8 * NSH, o[i]);
        }
        const v4u zz = {0, 0, 0, 0};
        const unsigned lo16 = lane * 16;
        asm volatile("buffer_store_dwordx4 %0, %1, %2, 0 offen sc0 sc1\n\tbuffer_store_dwordx4 %0, %1, %2, 0 offen offset:1024 sc0 sc1\n\t"
                     "buffer_store_dwordx4 %0, %1, %2, 0 offen offset:2048 sc0 sc1\n\ts_nop 1" :: "v"(zz), "v"(lo16), "s"(rs) : "memory");
        if (nby > 3072)
          asm volatile("buffer_store_dwordx4 %0, %1, %2, 0 offen offset:3072 sc0 sc1\n\tbuffer_store_dwordx4 %0, %3, %2, 0 offen sc0 sc1\n\ts_nop 1"
                       :: "v"(zz), "v"(lo16), "s"(rs), "v"(lo16 + 4096) : "memory");
      }
      if (dead) __hip_atomic_store(tmo, call, __ATOMIC_RELAXED, __HIP_MEMORY_SCOPE_AGENT);
      STAMP(6);
    }
  } else {
    float sxv = *(const CONST_AS float*)p.sx, swv = *(const CONST_AS float*)p.sw;
    unsigned char* pws = p.ws[peer & (NR - 1)];
    float* pout = p.out;
    asm volatile("" : "+s"(sxv), "+s"(swv), "+s"(pws), "+s"(pout));
    if (call == 0) call = __builtin_amdgcn_readfirstlane(__hip_atomic_load(cntp, __ATOMIC_RELAXED, __HIP_MEMORY_SCOPE_AGENT)) + 1;
    if (wid == 0) *(LDS_AS unsigned*)(size_t)(lds0 + LDS_CALL) = call;
    const float scale = sxv * swv;
    const int r = lane & 31, h = lane >> 5;
    const int aoff = r * 128 + (((2 * h) ^ ((r >> 1) & 7)) << 4);
    const int bkr = 32 * h + ((lane & 15) >> 1);
    const int boff = A_BYTES + bkr * TN + (((wid * 2 + ((lane >> 4) & 1)) ^ (((bkr >> 1) & 3) << 1)) << 4) + 8 * (lane & 1);
    struct Frag { v8i a[2][2], b[2]; };
    auto ldfrag = [&](Frag& f, int slotidx) {
      const unsigned char* slot = smem + slotidx * ST_BYTES;
#pragma unroll
      for (int kk = 0; kk < 2; ++kk) {
        const unsigned char* bp = slot + boff + kk * 64 * TN;
        v2i b0 = __builtin_amdgcn_ds_read_tr8_b64_v2i32((LDS_AS v2i*)(bp));
        v2i b1 = __builtin_amdgcn_ds_read_tr8_b64_v2i32((LDS_AS v2i*)(bp + 8 * TN));
        v2i b2 = __builtin_amdgcn_ds_read_tr8_b64_v2i32((LDS_AS v2i*)(bp + 16 * TN));
        v2i b3 = __builtin_amdgcn_ds_read_tr8_b64_v2i32((LDS_AS v2i*)(bp + 24 * TN));
        v4i l = __builtin_shufflevector(b0, b1, 0, 1, 2, 3), hh = __builtin_shufflevector(b2, b3, 0, 1, 2, 3);
        f.b[kk] = __builtin_shufflevector(l, hh, 0, 1, 2, 3, 4, 5, 6, 7);
        const int o = aoff ^ (kk << 6);
#pragma unroll
        for (int i = 0; i < 2; ++i) {
          v4i lo = *(const v4i*)(slot + o + i * 4096), hi = *(const v4i*)(slot + (o ^ 16) + i * 4096);
          f.a[kk][i] = __builtin_shufflevector(lo, hi, 0, 1, 2, 3, 4, 5, 6, 7);
        }
      }
    };
    v16f acc[2] = {};
    Frag f0, f1;
    int sl = 0;
    auto step = [&](Frag& cur, Frag& nxt) {
      __builtin_amdgcn_s_waitcnt(0xC07F);
      __builtin_amdgcn_sched_barrier(0);
      __builtin_amdgcn_s_barrier();
      asm volatile("" ::: "memory");
      __builtin_amdgcn_sched_barrier(0);
      sl = sl + 1 == NST ? 0 : sl + 1;
      ldfrag(nxt, sl);
#pragma unroll
      for (int kk = 0; kk < 2; ++kk)
#pragma unroll
        for (int i = 0; i < 2; ++i)
          acc[i] = __builtin_amdgcn_mfma_scale_f32_32x32x64_f8f6f4(cur.b[kk], cur.a[kk][i], acc[i], 1, 0, 0, 0, 0, 0);
#pragma unroll
      for (int i = 0; i < 4; ++i) { __builtin_amdgcn_sched_group_barrier(0x8, 1, 0); __builtin_amdgcn_sched_group_barrier(0x100, 4, 0); }
      __builtin_amdgcn_sched_barrier(0);
    };
    __builtin_amdgcn_s_barrier();
    asm volatile("" ::: "memory");
    STAMP(2);
    ldfrag(f0, 0);
#pragma unroll 1
    for (int g = 0; g < NKT; g += 2) { step(f0, f1); step(f1, f0); }
    STAMP(3);
    if (!roleY) {
      const bool bad = roleL && __builtin_amdgcn_ballot_w64(*(const LDS_AS unsigned*)(size_t)(lds0 + LDS_DEAD + (lane & 7) * 4) != 0) != 0;
      const int orow = roleL ? srcL * MS_ + YROWS + ti * TM : rank * MS_ + ti * TM;
      unsigned char* cb = smem + wid * 2 * CBUF_BYTES;
      const float sc2 = bad ? __builtin_nanf("") : scale;
#pragma unroll
      for (int i = 0; i < 2; ++i)
#pragma unroll
        for (int q = 0; q < 4; ++q) {
          v4u o;
#pragma unroll
          for (int j = 0; j < 4; ++j) { float v = acc[i][q * 4 + j] * sc2; o[j] = !(v <= 0.f) ? __float_as_uint(v) : 0u; }
          *(v4u*)(cb + (i * 32 + r) * 128 + (((q * 2 + h) ^ (r & 7)) << 4)) = o;
        }
      float* dst = pout + (size_t)(orow + (lane >> 3)) * NSH + tj * TN + wid * 32 + (lane & 7) * 4;
      v4u o[8];
#pragma unroll
      for (int i = 0; i < 8; ++i) o[i] = *(const v4u*)(cb + i * 1024 + (lane >> 3) * 128 + (((lane & 7) ^ (lane >> 3)) << 4));
#pragma unroll
      for (int i = 0; i < 8; ++i) st16(dst + i * 8 * NSH, o[i]);
      STAMP(4);
    } else {
      unsigned char* dbase = pws + WS_SLOT + ((call & 1) * NR + rank) * SLOT_SRC + (size_t)(ti * 32 + tj * 4 + wid) * CHUNK_BYTES;
      unsigned nbys[2], off = 0;
      auto piece = [&](int blk, unsigned nby) {
        const unsigned end = nby - off <= YPIECE + YTAIL ? nby : off + YPIECE;
        const v4i rs = make_rsrc(dbase + blk * 16 * CHUNK_BYTES, end);
        const unsigned o = off + lane * 16;
        v4u v = *(const v4u*)(smem + (wid * 2 + blk) * CBUF_BYTES + o);
        asm volatile("s_nop 4\n\tbuffer_store_dwordx4 %0, %1, %2, 0 offen sc0 sc1\n\ts_nop 1" :: "v"(v), "v"(o), "s"(rs) : "memory");
        off = end;
      };
#pragma unroll
      for (int blk = 0; blk < 2; ++blk) {
        unsigned* cb = (unsigned*)(smem + (wid * 2 + blk) * CBUF_BYTES);
        const unsigned cb_lds = lds0 + (wid * 2 + blk) * CBUF_BYTES;
        float u[16];
        unsigned msk = 0;
#pragma unroll
        for (int e = 0; e < 16; ++e) u[e] = acc[blk][e] * scale;
#define CA(n) "v_cmp_nge_f32 vcc, 0, %" #n "\n\tv_addc_co_u32 %0, vcc, %0, %0, vcc\n\t"
        asm volatile(CA(16) CA(15) CA(14) CA(13) CA(12) CA(11) CA(10) CA(9) CA(8) CA(7) CA(6) CA(5) CA(4) CA(3) CA(2) CA(1)
            : "+v"(msk) : "v"(u[0]), "v"(u[1]), "v"(u[2]), "v"(u[3]), "v"(u[4]), "v"(u[5]), "v"(u[6]), "v"(u[7]),
              "v"(u[8]), "v"(u[9]), "v"(u[10]), "v"(u[11]), "v"(u[12]), "v"(u[13]), "v"(u[14]), "v"(u[15]) : "vcc");
#undef CA
        if (msk == 0) u[0] = -0.0f;
        msk = max(msk, 1u);
        const unsigned pc = __builtin_popcount(msk);
        const unsigned inc = wave_scan(pc);
        *(LDS_AS unsigned short*)(size_t)(cb_lds + lane * 2) = (unsigned short)msk;
        unsigned a = cb_lds + 4 * (32 + inc - pc), t0_, t1_;
        const unsigned dump = cb_lds + CHUNK_BYTES + 16 + r * 4;
#define PKE(e, n) "v_bfe_i32 %1, %3, " #e ", 1\n\tv_bfi_b32 %2, %1, %0, %4\n\tv_mad_i32_i24 %0, %1, -4, %0\n\tds_write_b32 %2, %" #n "\n\t"
        asm volatile(PKE(0, 5) PKE(1, 6) PKE(2, 7) PKE(3, 8) PKE(4, 9) PKE(5, 10) PKE(6, 11) PKE(7, 12)
                     PKE(8, 13) PKE(9, 14) PKE(10, 15) PKE(11, 16) PKE(12, 17) PKE(13, 18) PKE(14, 19) PKE(15, 20)
            : "+v"(a), "=&v"(t0_), "=&v"(t1_) : "v"(msk), "v"(dump), "v"(u[0]), "v"(u[1]), "v"(u[2]), "v"(u[3]), "v"(u[4]), "v"(u[5]), "v"(u[6]), "v"(u[7]),
              "v"(u[8]), "v"(u[9]), "v"(u[10]), "v"(u[11]), "v"(u[12]), "v"(u[13]), "v"(u[14]), "v"(u[15]) : "memory");
#undef PKE
        const unsigned cnt = 32 + __builtin_amdgcn_readlane(inc, 63);
        if (lane < 3) cb[cnt + lane] = 1u;
        nbys[blk] = (cnt * 4 + 15) & ~15u;
        if (blk == 0) {
          piece(0, nbys[0]);
          STAMP(4);
          __builtin_amdgcn_sched_barrier(0);
        }
      }
#pragma unroll 1
      for (int blk = 0; blk < 2; ++blk) {
        const unsigned nby = blk ? nbys[1] : nbys[0];
        while (off < nby) { WAITV(0); piece(blk, nby); }
        off = 0;
      }
      STAMP(5);
    }
    if (tid == 0) __hip_atomic_store(cntp, call, __ATOMIC_RELAXED, __HIP_MEMORY_SCOPE_AGENT);
  }
#endif
}

extern "C" void kernel_launch(int rank, int nranks, ncclComm_t comm, hipStream_t stream,
                              void* const* d_in, const int* in_sizes, int n_in,
                              void* d_out, int out_size, const int* in_dtypes, int out_dtype,
                              void* const* d_out_all, void* const* d_ws_all, size_t ws_bytes) {
  if (nranks != NR || ws_bytes < WS_NEED || (unsigned)rank >= (unsigned)NR || n_in < 4 ||
      in_sizes[0] != MS_ * KD || in_sizes[1] != KD * ND || out_size != NR * MS_ * NSH) return;
  static bool init = false;
  if (!init) {
    HIP_CHECK(hipFuncSetAttribute((const void*)fused_gemm_a2a, hipFuncAttributeMaxDynamicSharedMemorySize, LDS_BYTES));
    init = true;
  }
  Params p;
  p.x = (const unsigned char*)d_in[0]; p.w = (const unsigned char*)d_in[1];
  p.sx = (const float*)d_in[2]; p.sw = (const float*)d_in[3];
  p.out = (float*)d_out;
  for (int i = 0; i < NR; ++i) p.ws[i] = (unsigned char*)d_ws_all[i];
  p.myws = p.ws[rank];
  p.rank = rank; p.pad = 0;
  fused_gemm_a2a<<<GRID, 512, LDS_BYTES, stream>>>(p);
  HIP_CHECK(hipGetLastError());
}
```

```cpp
#include <hip/hip_runtime.h>
#include <rccl/rccl.h>
#include <cstdio>
#include <utility>

#define HIP_CHECK(expr) do { hipError_t _e = (expr); if (_e != hipSuccess) { \
    fprintf(stderr, "HIP error %s at %s:%d\n", hipGetErrorString(_e), __FILE__, __LINE__); \
    return; } } while (0)

typedef int v2i __attribute__((ext_vector_type(2)));
typedef int v4i __attribute__((ext_vector_type(4)));
typedef unsigned v4u __attribute__((ext_vector_type(4)));
typedef int v8i __attribute__((ext_vector_type(8)));
typedef float v16f __attribute__((ext_vector_type(16)));
#define LDS_AS __attribute__((address_space(3)))
#define CONST_AS __attribute__((address_space(4)))
#define GLB_AS __attribute__((address_space(1)))
typedef GLB_AS unsigned gu32;

constexpr int NR = 8;
constexpr int MS_ = 512;
constexpr int KD = 4096, ND = 4096;
constexpr int NSH = 512;
constexpr int TM = 64, TN = 128, BK = 128;
constexpr int NST = 5;
constexpr int YROWS = 384, XROWS = MS_ - YROWS;
constexpr int XWIN = 4;
constexpr int XTAILW = 8;
constexpr int XEARLY = 4;
constexpr unsigned YPIECE = 768, YTAIL = 128;
constexpr int A_BYTES = TM * BK, B_BYTES = BK * TN, ST_BYTES = A_BYTES + B_BYTES;
constexpr int PPL = 6;
constexpr int NKT = KD / BK;
constexpr int CHUNK_BYTES = 128 + 4096;
constexpr int RBUF_BYTES = CHUNK_BYTES + 256;
constexpr int CBUF_BYTES = CHUNK_BYTES + 16 + 128;
constexpr int LDS_DEAD = NST * ST_BYTES;
constexpr int LDS_XFL = LDS_DEAD + 32;
constexpr int LDS_CALL = LDS_XFL + 32;
constexpr int LDS_BYTES = LDS_CALL + 16;
static_assert(LDS_BYTES <= 160 * 1024, "lds");
static_assert(8 * CBUF_BYTES + 4 * RBUF_BYTES + 1024 <= LDS_DEAD, "epilogue buffers live in the dead ring");
static_assert(XEARLY >= 4 && XEARLY < 8 && YPIECE + YTAIL <= 1024, "");
constexpr int GRID = 256;
constexpr unsigned long long DEADLINE_TICKS = 50000000ull;
constexpr size_t WS_CNT = 0;
constexpr size_t WS_TMO = GRID * 128;
constexpr size_t WS_XFLAG = WS_TMO + 128;
constexpr size_t WS_XS = 512 * 1024;
constexpr size_t XS_SRC = (size_t)XROWS * KD;
constexpr size_t WS_SLOT = WS_XS + 2 * NR * XS_SRC;
constexpr size_t SLOT_SRC = (YROWS / 32) * 16 * (size_t)CHUNK_BYTES;
constexpr size_t WS_NEED = WS_SLOT + 2 * NR * SLOT_SRC;
static_assert(WS_XFLAG + NR * 16 * 4 * 128 <= WS_XS, "");

struct Params {
  const unsigned char* x; const unsigned char* w; unsigned char* myws; int rank; int wcls;
  const float* sx; const float* sw; float* out;
  unsigned char* ws[NR];
};

struct XSched { int wait, flag; };
constexpr int xflag_at(int i) { return i - XWIN == 4 * XEARLY ? XEARLY : 0; }
constexpr XSched xsched(int i) {
  int pos = 8, ldpos[32] = {}, stpos[32] = {};
  for (int j = 0; j < 8; ++j) ldpos[j] = j;
  for (int j = 0; j < i; ++j) {
    if (xflag_at(j)) ++pos;
    stpos[j] = pos++;
    if ((j & 3) == 0 && j / 4 + 2 < 8) for (int q = 0; q < 4; ++q) ldpos[(j / 4 + 2) * 4 + q] = pos++;
  }
  int need = ldpos[i];
  if (i - XWIN - 1 >= 0 && i < 32 - XTAILW && stpos[i - XWIN - 1] > need) need = stpos[i - XWIN - 1];
  return XSched{pos - 1 - need, xflag_at(i)};
}
static_assert(4 * XEARLY + XWIN < 32 - XTAILW, "the early progress word is published behind a throttled wait");

template<int N, class F, int... I> __device__ __forceinline__ void static_for_impl(F&& f, std::integer_sequence<int, I...>) { (f(std::integral_constant<int, I>{}), ...); }
template<int N, class F> __device__ __forceinline__ void static_for(F&& f) { static_for_impl<N>(f, std::make_integer_sequence<int, N>{}); }

__device__ __forceinline__ unsigned wave_scan(unsigned v) {
#if defined(__HIP_DEVICE_COMPILE__)
  v += __builtin_amdgcn_update_dpp(0u, v, 0x111, 0xf, 0xf, true);
  v += __builtin_amdgcn_update_dpp(0u, v, 0x112, 0xf, 0xf, true);
  v += __builtin_amdgcn_update_dpp(0u, v, 0x114, 0xf, 0xf, true);
  v += __builtin_amdgcn_update_dpp(0u, v, 0x118, 0xf, 0xf, true);
  v += __builtin_amdgcn_update_dpp(0u, v, 0x142, 0xa, 0xf, false);
  v += __builtin_amdgcn_update_dpp(0u, v, 0x143, 0xc, 0xf, false);
#endif
  return v;
}
#define WAITV(n) asm volatile("s_waitcnt vmcnt(%0)" :: "n"(n) : "memory")
__device__ __forceinline__ void st16(void* ptr, v4u val) {
#if defined(__HIP_DEVICE_COMPILE__)
  asm volatile("global_store_dwordx4 %0, %1, off sc0 sc1\n\ts_nop 1" :: "v"(ptr), "v"(val) : "memory");
#endif
}
template<int KB> __device__ __forceinline__ void x_ldblk(v4u (&d)[32], const unsigned char* sp) {
#if defined(__HIP_DEVICE_COMPILE__)
  asm volatile("global_load_dwordx4 %0, %4, off offset:%6\n\tglobal_load_dwordx4 %1, %5, off offset:%6\n\t"
               "global_load_dwordx4 %2, %7, off offset:%6\n\tglobal_load_dwordx4 %3, %8, off offset:%6"
               : "=&v"(d[KB * 4]), "=&v"(d[KB * 4 + 1]), "=&v"(d[KB * 4 + 2]), "=&v"(d[KB * 4 + 3])
               : "v"(sp), "v"(sp + 2 * KD), "n"(KB * 512), "v"(sp + 4 * KD), "v"(sp + 6 * KD) : "memory");
#endif
}
template<int N, class T> __device__ __forceinline__ void vm_wait(T& v) {
#if defined(__HIP_DEVICE_COMPILE__)
  asm volatile("s_waitcnt vmcnt(%1)" : "+v"(v) : "n"(N) : "memory");
#endif
}
__device__ __forceinline__ v4i make_rsrc(const void* base, unsigned bytes) {
  return (v4i){(int)(size_t)base, (int)(((size_t)base >> 32) & 0xffff), (int)bytes, 0x00020000};
}

__global__ void __launch_bounds__(512, 1) fused_gemm_a2a(Params p) {
#if defined(__HIP_DEVICE_COMPILE__)
  extern __shared__ __attribute__((aligned(16))) unsigned char smem[];
  const int tid = threadIdx.x, lane = tid & 63;
  const int wid_all = __builtin_amdgcn_readfirstlane(tid >> 6);
  const int wid = wid_all & 3;
  const int b = blockIdx.x, xcd = b & 7, idx = b >> 3;
  int rank = p.rank;
  unsigned char* myws = p.myws;
  const unsigned char* px = p.x; const unsigned char* pw_ = p.w;
  int wcls = p.wcls;
  __builtin_amdgcn_sched_barrier(0);
  const bool roleS = xcd == 0, roleY = !roleS && (unsigned)(idx - 4) < 24u, roleL = !roleS && !roleY;
  const bool is_loader = (wid_all < 4) != roleL;
  const int yi = idx - 4;
  const int ti = roleS ? idx >> 2 : (roleY ? yi >> 2 : idx & 1);
  int rot = (roleS ? 4 * ti : (int)(0x1B15100B0500ull >> (8 * ti))) + 4 * xcd;
  int voffA[2], voffB[4];
#pragma unroll
  for (int i = 0; i < 2; ++i) { int r = (wid * 2 + i) * 8 + (lane >> 3); voffA[i] = (ti * TM + r) * KD + (((lane & 7) ^ ((r >> 1) & 7)) << 4) - i * 1024; }
#pragma unroll
  for (int i = 0; i < 4; ++i) { int kr = (wid * 4 + i) * 8 + (lane >> 3); voffB[i] = kr * ND + (((lane & 7) ^ (((kr >> 1) & 3) << 1)) << 4) - i * 1024; }
  asm volatile("" : "+v"(voffA[0]), "+v"(voffA[1]), "+v"(voffB[0]), "+v"(voffB[1]), "+v"(voffB[2]), "+v"(voffB[3]));
  asm volatile("" : "+s"(rot));
  asm volatile("" : "+s"(rank), "+s"(myws), "+s"(px), "+s"(pw_), "+s"(wcls));
  gu32* cntp = (gu32*)(myws + WS_CNT + b * 128);
  const unsigned peer = (unsigned)(xcd + rank) & (NR - 1);
  const unsigned srcL = (unsigned)(rank - xcd) & (NR - 1);
  const int tj = (roleL ? (idx >> 1) - wcls : idx) & 3;
  const int n0 = (roleY ? peer : rank) * NSH + tj * TN;
  const unsigned lds0 = (unsigned)(size_t)(LDS_AS unsigned char*)smem;
  gu32* tmo = (gu32*)(myws + WS_TMO);
  unsigned call = 0;

  auto give_up = [&](unsigned long long t0) -> bool {
    const unsigned long long dt = __builtin_amdgcn_s_memrealtime() - t0;
    if (dt < 2000) return false;
    if (dt > DEADLINE_TICKS) return true;
    __builtin_amdgcn_s_sleep(64);
    unsigned tv = 0;
    if (lane == 0) tv = __hip_atomic_load(tmo, __ATOMIC_RELAXED, __HIP_MEMORY_SCOPE_AGENT);
    return __builtin_amdgcn_readfirstlane(tv) == call;
  };
  gu32* xfl = (gu32*)(myws + WS_XFLAG) + ((srcL * 16 + ti * 8 + wid * 2 + (lane & 1)) * 4 + tj) * 32;
  auto wait_xflags = [&](unsigned nblk, int slot) {
    const unsigned long long t0 = __builtin_amdgcn_s_memrealtime();
    const unsigned want = 8 * (call - 1) + nblk;
    unsigned fv = want, dd = 0;
    for (;;) {
      asm volatile("s_mov_b64 exec, 3\n\tglobal_load_dword %0, %1, off sc0 sc1\n\ts_mov_b64 exec, -1\n\ts_waitcnt vmcnt(0)"
                   : "+v"(fv) : "v"(xfl) : "memory");
      if (__builtin_amdgcn_ballot_w64(fv < want) == 0) break;
      if (give_up(t0)) {
        __hip_atomic_store(tmo, call, __ATOMIC_RELAXED, __HIP_MEMORY_SCOPE_AGENT);
        WAITV(0);
        dd = 1;
        break;
      }
      __builtin_amdgcn_s_sleep(8);
    }
    *(LDS_AS unsigned*)(size_t)(lds0 + LDS_DEAD + slot * 4) = dd;
    __builtin_amdgcn_s_waitcnt(0xC07F);
  };

  if (__builtin_expect(roleL && !is_loader, 0)) {
    if (idx < 4 && peer < (unsigned)NR) {
      const int xo = (idx * 32 + wid * 8 + (lane >> 5)) * KD + (lane & 31) * 16;
      const unsigned char* sp = px + (size_t)YROWS * KD + xo;
      v4u d[32];
      unsigned cntv;
      asm volatile("global_load_dword %0, %1, off sc1" : "=v"(cntv) : "v"(cntp) : "memory");
      x_ldblk<0>(d, sp); x_ldblk<1>(d, sp);
      unsigned char* pw = p.ws[peer & (NR - 1)];
      asm volatile("s_waitcnt vmcnt(8)\n\tv_readfirstlane_b32 %0, %1" : "=s"(call) : "v"(cntv) : "memory");
      call += 1;
      unsigned char* dp = pw + WS_XS + ((call & 1) * NR + rank) * XS_SRC + xo;
      gu32* pf = (gu32*)(pw + WS_XFLAG) + ((rank * 16 + idx * 4 + wid) * 4 + (lane & 3)) * 32;
      const unsigned fbase = 8 * (call - 1);
      static_for<32>([&](auto ic) {
        constexpr int i = decltype(ic)::value;
        constexpr XSched xs = xsched(i);
        vm_wait<xs.wait>(d[i]);
        if (xs.flag && lane < 4) __hip_atomic_store(pf, fbase + xs.flag, __ATOMIC_RELAXED, __HIP_MEMORY_SCOPE_SYSTEM);
        st16(dp + (i >> 2) * 512 + (i & 3) * 2 * KD, d[i]);
        if constexpr ((i & 3) == 0 && i / 4 + 2 < 8) x_ldblk<i / 4 + 2>(d, sp);
      });
      WAITV(0);
      if (lane < 4) __hip_atomic_store(pf, fbase + 8, __ATOMIC_RELAXED, __HIP_MEMORY_SCOPE_SYSTEM);
    }
  }

  if (__builtin_expect(is_loader, 1)) {
    v4i rx = make_rsrc(px, MS_ * KD);
    const v4i rw = make_rsrc(pw_ + n0, KD * ND - n0);
    auto issue = [&](const bool L, int kt, unsigned slot) __attribute__((always_inline)) {
      const int soA = kt * BK, soB = kt * (BK * ND);
      if (L)
        asm volatile("s_mov_b32 m0, %0\n\ts_nop 0\n\tbuffer_load_dwordx4 %1, %3, %4 offen sc0 sc1 lds\n\tbuffer_load_dwordx4 %2, %3, %4 offen offset:1024 sc0 sc1 lds"
                     :: "s"(slot + wid * 2048), "v"(voffA[0]), "v"(voffA[1]), "s"(rx), "s"(soA) : "memory");
      else
        asm volatile("s_mov_b32 m0, %0\n\ts_nop 0\n\tbuffer_load_dwordx4 %1, %3, %4 offen lds\n\tbuffer_load_dwordx4 %2, %3, %4 offen offset:1024 lds"
                     :: "s"(slot + wid * 2048), "v"(voffA[0]), "v"(voffA[1]), "s"(rx), "s"(soA) : "memory");
      asm volatile("s_mov_b32 m0, %0\n\ts_nop 0\n\t"
                   "buffer_load_dwordx4 %1, %5, %6 offen lds\n\t"
                   "buffer_load_dwordx4 %2, %5, %6 offen offset:1024 lds\n\t"
                   "buffer_load_dwordx4 %3, %5, %6 offen offset:2048 lds\n\t"
                   "buffer_load_dwordx4 %4, %5, %6 offen offset:3072 lds"
                   :: "s"(slot + A_BYTES + wid * 4096), "v"(voffB[0]), "v"(voffB[1]), "v"(voffB[2]), "v"(voffB[3]), "s"(rw), "s"(soB) : "memory");
    };
    auto ring = [&](const bool L) __attribute__((always_inline)) {
      auto kstage = [&](int g) { return L ? (g < 16 ? g ^ (ti << 3) : g) : ((g + rot) & (NKT - 1)); };
      static_assert(NST == 5, "prologue");
      issue(L, kstage(0), lds0); issue(L, kstage(1), lds0 + ST_BYTES);
      WAITV(PPL);
      __builtin_amdgcn_s_barrier();
      issue(L, kstage(2), lds0 + 2 * ST_BYTES); issue(L, kstage(3), lds0 + 3 * ST_BYTES);
      WAITV(2 * PPL);
      __builtin_amdgcn_s_barrier();
      issue(L, kstage(4), lds0 + 4 * ST_BYTES); issue(L, kstage(5), lds0);
      unsigned slot = lds0 + ST_BYTES;
#pragma unroll 1
      for (int g = NST + 1; g < NKT; ++g) {
        int kt = kstage(g);
        asm volatile("s_waitcnt vmcnt(%1)" : "+s"(kt) : "n"((NST - 2) * PPL) : "memory");
        __builtin_amdgcn_s_barrier();
        if (L && g == XEARLY * 4) {
          const unsigned fv = *(const LDS_AS unsigned*)(size_t)(lds0 + LDS_XFL + wid * 8 + (lane & 1) * 4);
          if (__builtin_amdgcn_ballot_w64(fv < 8 * call) != 0) wait_xflags(8, 4 + wid);
        }
        issue(L, kt, slot);
        if (L && g == XEARLY * 4 - (NST - 1))
          asm volatile("s_mov_b32 m0, %0\n\ts_mov_b64 exec, 3\n\tglobal_load_lds_dword %1, off sc0 sc1\n\ts_mov_b64 exec, -1"
                       :: "s"(lds0 + LDS_XFL + wid * 8), "v"(xfl) : "memory");
        slot = slot + ST_BYTES == lds0 + NST * ST_BYTES ? lds0 : slot + ST_BYTES;
      }
    };
    if (__builtin_expect(roleL, 0)) {
      call = __builtin_amdgcn_readfirstlane(__hip_atomic_load(cntp, __ATOMIC_RELAXED, __HIP_MEMORY_SCOPE_AGENT)) + 1;
      if (wid == 0) *(LDS_AS unsigned*)(size_t)(lds0 + LDS_CALL) = call - 1;
      rx = make_rsrc(myws + WS_XS + ((call & 1) * NR + srcL) * XS_SRC, XROWS * KD);
      *(LDS_AS unsigned*)(size_t)(lds0 + LDS_DEAD + 16 + wid * 4) = 0;
      wait_xflags(XEARLY, wid);
      ring(true);
    } else {
      if (wid == 0)
        asm volatile("s_mov_b32 m0, %0\n\ts_mov_b64 exec, 1\n\tglobal_load_lds_dword %1, off sc1\n\ts_mov_b64 exec, -1"
                     :: "s"(lds0 + LDS_CALL), "v"(cntp) : "memory");
      ring(false);
    }
    static_for<NST - 1>([&](auto ic) {
      constexpr int i = decltype(ic)::value;
      WAITV((NST - 2 - i) * PPL);
      __builtin_amdgcn_s_barrier();
    });
    __builtin_amdgcn_s_barrier();
    asm volatile("" ::: "memory");

    if (roleY) {
      call = __builtin_amdgcn_readfirstlane(*(const LDS_AS unsigned*)(size_t)(lds0 + LDS_CALL)) + 1;
      const int id = ((xcd - 1) * 24 + yi) * 4 + wid;
      const int si = id / 96, c96 = id - si * 96;
      const unsigned src = (unsigned)(rank + 1 + si) & (NR - 1);
      const int cbk = c96 & 15;
      unsigned* rb = (unsigned*)(smem + 8 * CBUF_BYTES + wid * RBUF_BYTES);
      unsigned* tab = rb + CHUNK_BYTES / 4;
      const unsigned rb_lds = lds0 + 8 * CBUF_BYTES + wid * RBUF_BYTES;
      const unsigned long long t0 = __builtin_amdgcn_s_memrealtime();
      bool dead = false;
#pragma unroll 1
      for (int blk = 0; blk < 2; ++blk) {
        const int rb32 = (c96 >> 4) * 2 + blk;
        unsigned char* sp0 = myws + WS_SLOT + ((call & 1) * NR + src) * SLOT_SRC + (size_t)(rb32 * 16 + cbk) * CHUNK_BYTES;
        v4u hd = {~0u, ~0u, ~0u, ~0u};
        for (;;) {
          asm volatile("s_mov_b64 exec, 0xff\n\tglobal_load_dwordx4 %0, %1, off sc0 sc1\n\ts_mov_b64 exec, -1\n\ts_waitcnt vmcnt(0)" : "+v"(hd) : "v"(sp0 + lane * 16) : "memory");
          const unsigned lo = min(min(hd[0] & 0xffffu, hd[1] & 0xffffu), min(hd[2] & 0xffffu, hd[3] & 0xffffu));
          const unsigned hi = min(min(hd[0], hd[1]), min(hd[2], hd[3]));
          if (__builtin_amdgcn_ballot_w64(lo == 0 || hi < 0x10000u) == 0) break;
          if (dead || give_up(t0)) { dead = true; break; }
          __builtin_amdgcn_s_sleep(4);
        }
        if (lane < 8) *(v4u*)(rb + lane * 4) = hd;
        const unsigned msk = ((const unsigned short*)rb)[lane];
        const unsigned pc = __builtin_popcount(msk);
        const unsigned inc = wave_scan(pc);
        const unsigned nby = (128 + 4 * __builtin_amdgcn_readlane(inc, 63) + 15) & ~15u;
        const v4i rs = make_rsrc(sp0, nby);
        v4u d[4];
        unsigned of[4];
#pragma unroll
        for (int i = 0; i < 4; ++i) { of[i] = 128 + i * 1024u + lane * 16u; d[i] = (v4u){0, 0, 0, 0}; }
        unsigned pend = (nby > 128 + 3072) ? 15u : (nby > 128 + 2048) ? 7u : (nby > 128 + 1024) ? 3u : 1u;
        auto fetch = [&]() {
#pragma unroll
          for (int i = 0; i < 4; ++i)
            if (pend >> i & 1) asm volatile("s_nop 4\n\tbuffer_load_dwordx4 %0, %1, %2, 0 offen sc0 sc1" : "+v"(d[i]) : "v"(of[i]), "s"(rs) : "memory");
        };
        fetch();
        tab[lane] = ((32 + inc - pc) << 16) | msk;
        unsigned ga[4][4], gm[4][4];
#pragma unroll
        for (int i = 0; i < 4; ++i) {
          const unsigned t = tab[i * 8 + (lane >> 3) + 32 * (lane & 1)];
          const unsigned sh = (lane & 6) * 2;
          const unsigned a = rb_lds + 4 * ((t >> 16) + __builtin_popcount(t & ((1u << sh) - 1u)));
          const unsigned nib = t >> sh;
          ga[i][0] = a;
          asm("v_bfe_i32 %0, %7, 0, 1\n\tv_bfe_i32 %1, %7, 1, 1\n\tv_bfe_i32 %2, %7, 2, 1\n\tv_bfe_i32 %3, %7, 3, 1\n\t"
              "v_mad_i32_i24 %4, %0, -4, %8\n\tv_mad_i32_i24 %5, %1, -4, %4\n\tv_mad_i32_i24 %6, %2, -4, %5"
              : "=&v"(gm[i][0]), "=&v"(gm[i][1]), "=&v"(gm[i][2]), "=&v"(gm[i][3]), "=&v"(ga[i][1]), "=&v"(ga[i][2]), "=&v"(ga[i][3]) : "v"(nib), "v"(a));
        }
        for (;;) {
          asm volatile("s_waitcnt vmcnt(0)" : "+v"(d[0]), "+v"(d[1]), "+v"(d[2]), "+v"(d[3]) :: "memory");
#pragma unroll
          for (int i = 0; i < 4; ++i)
            if (pend >> i & 1) {
              const bool miss = of[i] < nby && min(min(d[i][0], d[i][1]), min(d[i][2], d[i][3])) == 0;
              if (__builtin_amdgcn_ballot_w64(miss) == 0) { pend &= ~(1u << i); *(v4u*)((unsigned char*)rb + of[i]) = d[i]; }
            }
          if (pend == 0 || dead) break;
          if (give_up(t0)) { dead = true; break; }
          fetch();
        }
        float* dst = p.out + (size_t)(src * MS_ + rb32 * 32 + (lane >> 3)) * NSH + cbk * 32 + (lane & 7) * 4;
        const unsigned nanv = dead ? 0x7fc00000u : 0u;
        v4u o[4];
#pragma unroll
        for (int i = 0; i < 4; ++i)
#pragma unroll
          for (int j = 0; j < 4; ++j) o[i][j] = *(const LDS_AS unsigned*)(size_t)ga[i][j];
#pragma unroll
        for (int i = 0; i < 4; ++i) {
#pragma unroll
          for (int j = 0; j < 4; ++j) o[i][j] = (o[i][j] & gm[i][j]) | nanv;
          st16(dst + i * 8 * NSH, o[i]);
        }
        const v4u zz = {0, 0, 0, 0};
        const unsigned lo16 = lane * 16;
        asm volatile("buffer_store_dwordx4 %0, %1, %2, 0 offen sc0 sc1\n\tbuffer_store_dwordx4 %0, %1, %2, 0 offen offset:1024 sc0 sc1\n\t"
                     "buffer_store_dwordx4 %0, %1, %2, 0 offen offset:2048 sc0 sc1\n\ts_nop 1" :: "v"(zz), "v"(lo16), "s"(rs) : "memory");
        if (nby > 3072)
          asm volatile("buffer_store_dwordx4 %0, %1, %2, 0 offen offset:3072 sc0 sc1\n\tbuffer_store_dwordx4 %0, %3, %2, 0 offen sc0 sc1\n\ts_nop 1"
                       :: "v"(zz), "v"(lo16), "s"(rs), "v"(lo16 + 4096) : "memory");
      }
      if (dead) __hip_atomic_store(tmo, call, __ATOMIC_RELAXED, __HIP_MEMORY_SCOPE_AGENT);
    }
  } else {
    float sxv = *(const CONST_AS float*)p.sx, swv = *(const CONST_AS float*)p.sw;
    unsigned char* pws = p.ws[peer & (NR - 1)];
    float* pout = p.out;
    asm volatile("" : "+s"(sxv), "+s"(swv), "+s"(pws), "+s"(pout));
    const float scale = sxv * swv;
    const int r = lane & 31, h = lane >> 5;
    const int aoff = r * 128 + (((2 * h) ^ ((r >> 1) & 7)) << 4);
    const int bkr = 32 * h + ((lane & 15) >> 1);
    const int boff = A_BYTES + bkr * TN + (((wid * 2 + ((lane >> 4) & 1)) ^ (((bkr >> 1) & 3) << 1)) << 4) + 8 * (lane & 1);
    struct Frag { v8i a[2][2], b[2]; };
    auto ldfrag = [&](Frag& f, int slotidx) {
      const unsigned char* slot = smem + slotidx * ST_BYTES;
#pragma unroll
      for (int kk = 0; kk < 2; ++kk) {
        const unsigned char* bp = slot + boff + kk * 64 * TN;
        v2i b0 = __builtin_amdgcn_ds_read_tr8_b64_v2i32((LDS_AS v2i*)(bp));
        v2i b1 = __builtin_amdgcn_ds_read_tr8_b64_v2i32((LDS_AS v2i*)(bp + 8 * TN));
        v2i b2 = __builtin_amdgcn_ds_read_tr8_b64_v2i32((LDS_AS v2i*)(bp + 16 * TN));
        v2i b3 = __builtin_amdgcn_ds_read_tr8_b64_v2i32((LDS_AS v2i*)(bp + 24 * TN));
        v4i l = __builtin_shufflevector(b0, b1, 0, 1, 2, 3), hh = __builtin_shufflevector(b2, b3, 0, 1, 2, 3);
        f.b[kk] = __builtin_shufflevector(l, hh, 0, 1, 2, 3, 4, 5, 6, 7);
        const int o = aoff ^ (kk << 6);
#pragma unroll
        for (int i = 0; i < 2; ++i) {
          v4i lo = *(const v4i*)(slot + o + i * 4096), hi = *(const v4i*)(slot + (o ^ 16) + i * 4096);
          f.a[kk][i] = __builtin_shufflevector(lo, hi, 0, 1, 2, 3, 4, 5, 6, 7);
        }
      }
    };
    v16f acc[2] = {};
    Frag f0, f1;
    int sl = 0;
    auto step = [&](Frag& cur, Frag& nxt) {
      __builtin_amdgcn_s_waitcnt(0xC07F);
      __builtin_amdgcn_sched_barrier(0);
      __builtin_amdgcn_s_barrier();
      asm volatile("" ::: "memory");
      __builtin_amdgcn_sched_barrier(0);
      sl = sl + 1 == NST ? 0 : sl + 1;
      ldfrag(nxt, sl);
#pragma unroll
      for (int kk = 0; kk < 2; ++kk)
#pragma unroll
        for (int i = 0; i < 2; ++i)
          acc[i] = __builtin_amdgcn_mfma_scale_f32_32x32x64_f8f6f4(cur.b[kk], cur.a[kk][i], acc[i], 1, 0, 0, 0, 0, 0);
#pragma unroll
      for (int i = 0; i < 4; ++i) { __builtin_amdgcn_sched_group_barrier(0x8, 1, 0); __builtin_amdgcn_sched_group_barrier(0x100, 4, 0); }
      __builtin_amdgcn_sched_barrier(0);
    };
    __builtin_amdgcn_s_barrier();
    asm volatile("" ::: "memory");
    call = __builtin_amdgcn_readfirstlane(*(const LDS_AS unsigned*)(size_t)(lds0 + LDS_CALL)) + 1;
    asm volatile("" : "+s"(call));
    ldfrag(f0, 0);
#pragma unroll 1
    for (int g = 0; g < NKT - 2; g += 2) { step(f0, f1); step(f1, f0); }
    step(f0, f1);
#pragma unroll
    for (int i = 0; i < 2; ++i)
#pragma unroll
      for (int kk = 0; kk < 2; ++kk)
        acc[i] = __builtin_amdgcn_mfma_scale_f32_32x32x64_f8f6f4(f1.b[kk], f1.a[kk][i], acc[i], 1, 0, 0, 0, 0, 0);
    __builtin_amdgcn_sched_barrier(0);
    __builtin_amdgcn_s_barrier();
    asm volatile("" ::: "memory");
    if (!roleY) {
      const bool bad = roleL && __builtin_amdgcn_ballot_w64(*(const LDS_AS unsigned*)(size_t)(lds0 + LDS_DEAD + (lane & 7) * 4) != 0) != 0;
      const int orow = roleL ? srcL * MS_ + YROWS + ti * TM : rank * MS_ + ti * TM;
      unsigned char* cb = smem + wid * 2 * CBUF_BYTES;
      const float sc2 = bad ? __builtin_nanf("") : scale;
      float* dst = pout + (size_t)(orow + (lane >> 3)) * NSH + tj * TN + wid * 32 + (lane & 7) * 4;
#pragma unroll
      for (int i = 0; i < 2; ++i) {
#pragma unroll
        for (int q = 0; q < 4; ++q) {
          v4u o;
#pragma unroll
          for (int j = 0; j < 4; ++j) o[j] = __float_as_uint(__builtin_elementwise_maximum(acc[i][q * 4 + j] * sc2, 0.f));
          *(v4u*)(cb + (i * 32 + r) * 128 + (((q * 2 + h) ^ (r & 7)) << 4)) = o;
        }
        v4u o[4];
#pragma unroll
        for (int q = 0; q < 4; ++q) o[q] = *(const v4u*)(cb + (i * 4 + q) * 1024 + (lane >> 3) * 128 + (((lane & 7) ^ (lane >> 3)) << 4));
#pragma unroll
        for (int q = 0; q < 4; ++q) st16(dst + (i * 4 + q) * 8 * NSH, o[q]);
      }
    } else {
      unsigned char* dbase = pws + WS_SLOT + ((call & 1) * NR + rank) * SLOT_SRC + (size_t)(ti * 32 + tj * 4 + wid) * CHUNK_BYTES;
      unsigned nbys[2], off = 0;
      auto piece = [&](int blk, unsigned nby) {
        const unsigned end = nby - off <= YPIECE + YTAIL ? nby : off + YPIECE;
        const v4i rs = make_rsrc(dbase + blk * 16 * CHUNK_BYTES, end);
        const unsigned o = off + lane * 16;
        v4u v = *(const v4u*)(smem + (wid * 2 + blk) * CBUF_BYTES + o);
        asm volatile("s_nop 4\n\tbuffer_store_dwordx4 %0, %1, %2, 0 offen sc0 sc1\n\ts_nop 1" :: "v"(v), "v"(o), "s"(rs) : "memory");
        off = end;
      };
#pragma unroll
      for (int blk = 0; blk < 2; ++blk) {
        unsigned* cb = (unsigned*)(smem + (wid * 2 + blk) * CBUF_BYTES);
        const unsigned cb_lds = lds0 + (wid * 2 + blk) * CBUF_BYTES;
        float u[16];
        unsigned msk = 0;
#pragma unroll
        for (int e = 0; e < 16; ++e) u[e] = acc[blk][e] * scale;
#define CA(n) "v_cmp_nge_f32 vcc, 0, %" #n "\n\tv_addc_co_u32 %0, vcc, %0, %0, vcc\n\t"
        asm volatile(CA(16) CA(15) CA(14) CA(13) CA(12) CA(11) CA(10) CA(9) CA(8) CA(7) CA(6) CA(5) CA(4) CA(3) CA(2) CA(1)
            : "+v"(msk) : "v"(u[0]), "v"(u[1]), "v"(u[2]), "v"(u[3]), "v"(u[4]), "v"(u[5]), "v"(u[6]), "v"(u[7]),
              "v"(u[8]), "v"(u[9]), "v"(u[10]), "v"(u[11]), "v"(u[12]), "v"(u[13]), "v"(u[14]), "v"(u[15]) : "vcc");
#undef CA
        if (msk == 0) u[0] = -0.0f;
        msk = max(msk, 1u);
        const unsigned pc = __builtin_popcount(msk);
        const unsigned inc = wave_scan(pc);
        *(LDS_AS unsigned short*)(size_t)(cb_lds + lane * 2) = (unsigned short)msk;
        unsigned a = cb_lds + 4 * (32 + inc - pc), t0_, t1_;
        const unsigned dump = cb_lds + CHUNK_BYTES + 16 + r * 4;
#define PKE(e, n) "v_bfe_i32 %1, %3, " #e ", 1\n\tv_bfi_b32 %2, %1, %0, %4\n\tv_mad_i32_i24 %0, %1, -4, %0\n\tds_write_b32 %2, %" #n "\n\t"
        asm volatile(PKE(0, 5) PKE(1, 6) PKE(2, 7) PKE(3, 8) PKE(4, 9) PKE(5, 10) PKE(6, 11) PKE(7, 12)
                     PKE(8, 13) PKE(9, 14) PKE(10, 15) PKE(11, 16) PKE(12, 17) PKE(13, 18) PKE(14, 19) PKE(15, 20)
            : "+v"(a), "=&v"(t0_), "=&v"(t1_) : "v"(msk), "v"(dump), "v"(u[0]), "v"(u[1]), "v"(u[2]), "v"(u[3]), "v"(u[4]), "v"(u[5]), "v"(u[6]), "v"(u[7]),
              "v"(u[8]), "v"(u[9]), "v"(u[10]), "v"(u[11]), "v"(u[12]), "v"(u[13]), "v"(u[14]), "v"(u[15]) : "memory");
#undef PKE
        const unsigned cnt = 32 + __builtin_amdgcn_readlane(inc, 63);
        if (lane < 3) cb[cnt + lane] = 1u;
        nbys[blk] = (cnt * 4 + 15) & ~15u;
        if (blk == 0) {
          piece(0, nbys[0]);
          __builtin_amdgcn_sched_barrier(0);
        }
      }
#pragma unroll 1
      for (int blk = 0; blk < 2; ++blk) {
        const unsigned nby = blk ? nbys[1] : nbys[0];
        while (off < nby) { WAITV(0); piece(blk, nby); }
        off = 0;
      }
    }
    if (wid == 0 && lane == 0) __hip_atomic_store(cntp, call, __ATOMIC_RELAXED, __HIP_MEMORY_SCOPE_AGENT);
  }
#endif
}

extern "C" void kernel_launch(int rank, int nranks, ncclComm_t comm, hipStream_t stream,
                              void* const* d_in, const int* in_sizes, int n_in,
                              void* d_out, int out_size, const int* in_dtypes, int out_dtype,
                              void* const* d_out_all, void* const* d_ws_all, size_t ws_bytes) {
  if (nranks != NR || ws_bytes < WS_NEED || (unsigned)rank >= (unsigned)NR || n_in < 4 ||
      in_sizes[0] != MS_ * KD || in_sizes[1] != KD * ND || out_size != NR * MS_ * NSH) return;
  static bool init = false;
  if (!init) {
    HIP_CHECK(hipFuncSetAttribute((const void*)fused_gemm_a2a, hipFuncAttributeMaxDynamicSharedMemorySize, LDS_BYTES));
    init = true;
  }
  Params p;
  p.x = (const unsigned char*)d_in[0]; p.w = (const unsigned char*)d_in[1];
  p.sx = (const float*)d_in[2]; p.sw = (const float*)d_in[3];
  p.out = (float*)d_out;
  for (int i = 0; i < NR; ++i) p.ws[i] = (unsigned char*)d_ws_all[i];
  p.myws = p.ws[rank];
  p.rank = rank; p.wcls = (int)(((size_t)p.w >> 7) & 3);
  fused_gemm_a2a<<<GRID, 512, LDS_BYTES, stream>>>(p);
  HIP_CHECK(hipGetLastError());
}
```

```cpp
#include <hip/hip_runtime.h>
#include <rccl/rccl.h>
#include <cstdio>
#include <utility>

#define HIP_CHECK(expr) do { hipError_t _e = (expr); if (_e != hipSuccess) { \
    fprintf(stderr, "HIP error %s at %s:%d\n", hipGetErrorString(_e), __FILE__, __LINE__); \
    return; } } while (0)

typedef int v2i __attribute__((ext_vector_type(2)));
typedef int v4i __attribute__((ext_vector_type(4)));
typedef unsigned v4u __attribute__((ext_vector_type(4)));
typedef int v8i __attribute__((ext_vector_type(8)));
typedef float v16f __attribute__((ext_vector_type(16)));
#define LDS_AS __attribute__((address_space(3)))
#define CONST_AS __attribute__((address_space(4)))
#define GLB_AS __attribute__((address_space(1)))
typedef GLB_AS unsigned gu32;

constexpr int NR = 8;
constexpr int MS_ = 512;
constexpr int KD = 4096, ND = 4096;
constexpr int NSH = 512;
constexpr int TM = 64, TN = 128, BK = 128;
constexpr int NST = 5;
constexpr int YROWS = 384, XROWS = MS_ - YROWS;
constexpr int XWIN = 3;
constexpr int XTAILW = 8;
constexpr int XEARLY = 4;
constexpr int XMID = 5;
constexpr unsigned YPIECE = 768, YTAIL = 128;
constexpr int A_BYTES = TM * BK, B_BYTES = BK * TN, ST_BYTES = A_BYTES + B_BYTES;
constexpr int PPL = 6;
constexpr int NKT = KD / BK;
constexpr int CHUNK_BYTES = 128 + 4096;
constexpr int RBUF_BYTES = CHUNK_BYTES + 256;
constexpr int CBUF_BYTES = CHUNK_BYTES + 16 + 128;
constexpr int LDS_DEAD = NST * ST_BYTES;
constexpr int LDS_XFL = LDS_DEAD + 32;
constexpr int LDS_CALL = LDS_XFL + 32;
constexpr int LDS_BYTES = LDS_CALL + 16;
static_assert(LDS_BYTES <= 160 * 1024, "lds");
static_assert(8 * CBUF_BYTES + 4 * RBUF_BYTES + 1024 <= LDS_DEAD, "epilogue buffers live in the dead ring");
static_assert(XEARLY >= 4 && XMID < 8 && YPIECE + YTAIL <= 1024, "");
constexpr int GRID = 256;
constexpr unsigned long long DEADLINE_TICKS = 50000000ull;
constexpr size_t WS_CNT = 0;
constexpr size_t WS_TMO = GRID * 128;
constexpr size_t WS_XFLAG = WS_TMO + 128;
constexpr size_t WS_XS = 512 * 1024;
constexpr size_t XS_SRC = (size_t)XROWS * KD;
constexpr size_t WS_SLOT = WS_XS + 2 * NR * XS_SRC;
constexpr size_t SLOT_SRC = (YROWS / 32) * 16 * (size_t)CHUNK_BYTES;
constexpr size_t WS_NEED = WS_SLOT + 2 * NR * SLOT_SRC;
static_assert(WS_XFLAG + NR * 16 * 4 * 128 <= WS_XS, "");

struct Params {
  const unsigned char* x; const unsigned char* w; unsigned char* myws; int rank; int wcls;
  const float* sx; const float* sw; float* out;
  unsigned char* ws[NR];
};

struct XSched { int wait, flag; };
constexpr int xflag_at(int i) { return i - XWIN == 4 * XEARLY ? XEARLY : (i - XWIN == 4 * XMID ? XMID : 0); }
constexpr XSched xsched(int i) {
  int pos = 8, ldpos[32] = {}, stpos[32] = {};
  for (int j = 0; j < 8; ++j) ldpos[j] = j;
  for (int j = 0; j < i; ++j) {
    if (xflag_at(j)) ++pos;
    stpos[j] = pos++;
    if ((j & 3) == 0 && j / 4 + 2 < 8) for (int q = 0; q < 4; ++q) ldpos[(j / 4 + 2) * 4 + q] = pos++;
  }
  int need = ldpos[i];
  if (i - XWIN - 1 >= 0 && i < 32 - XTAILW && stpos[i - XWIN - 1] > need) need = stpos[i - XWIN - 1];
  return XSched{pos - 1 - need, xflag_at(i)};
}
static_assert(4 * XMID + XWIN < 32 - XTAILW && XMID > XEARLY, "progress words are published behind throttled waits");

template<int N, class F, int... I> __device__ __forceinline__ void static_for_impl(F&& f, std::integer_sequence<int, I...>) { (f(std::integral_constant<int, I>{}), ...); }
template<int N, class F> __device__ __forceinline__ void static_for(F&& f) { static_for_impl<N>(f, std::make_integer_sequence<int, N>{}); }

__device__ __forceinline__ unsigned wave_scan(unsigned v) {
#if defined(__HIP_DEVICE_COMPILE__)
  v += __builtin_amdgcn_update_dpp(0u, v, 0x111, 0xf, 0xf, true);
  v += __builtin_amdgcn_update_dpp(0u, v, 0x112, 0xf, 0xf, true);
  v += __builtin_amdgcn_update_dpp(0u, v, 0x114, 0xf, 0xf, true);
  v += __builtin_amdgcn_update_dpp(0u, v, 0x118, 0xf, 0xf, true);
  v += __builtin_amdgcn_update_dpp(0u, v, 0x142, 0xa, 0xf, false);
  v += __builtin_amdgcn_update_dpp(0u, v, 0x143, 0xc, 0xf, false);
#endif
  return v;
}
#define WAITV(n) asm volatile("s_waitcnt vmcnt(%0)" :: "n"(n) : "memory")
__device__ __forceinline__ void st16(void* ptr, v4u val) {
#if defined(__HIP_DEVICE_COMPILE__)
  asm volatile("global_store_dwordx4 %0, %1, off sc0 sc1\n\ts_nop 1" :: "v"(ptr), "v"(val) : "memory");
#endif
}
template<int KB> __device__ __forceinline__ void x_ldblk(v4u (&d)[32], const unsigned char* sp) {
#if defined(__HIP_DEVICE_COMPILE__)
  asm volatile("global_load_dwordx4 %0, %4, off offset:%6\n\tglobal_load_dwordx4 %1, %5, off offset:%6\n\t"
               "global_load_dwordx4 %2, %7, off offset:%6\n\tglobal_load_dwordx4 %3, %8, off offset:%6"
               : "=&v"(d[KB * 4]), "=&v"(d[KB * 4 + 1]), "=&v"(d[KB * 4 + 2]), "=&v"(d[KB * 4 + 3])
               : "v"(sp), "v"(sp + 2 * KD), "n"(KB * 512), "v"(sp + 4 * KD), "v"(sp + 6 * KD) : "memory");
#endif
}
template<int N, class T> __device__ __forceinline__ void vm_wait(T& v) {
#if defined(__HIP_DEVICE_COMPILE__)
  asm volatile("s_waitcnt vmcnt(%1)" : "+v"(v) : "n"(N) : "memory");
#endif
}
__device__ __forceinline__ v4i make_rsrc(const void* base, unsigned bytes) {
  return (v4i){(int)(size_t)base, (int)(((size_t)base >> 32) & 0xffff), (int)bytes, 0x00020000};
}

__global__ void __launch_bounds__(512, 1) fused_gemm_a2a(Params p) {
#if defined(__HIP_DEVICE_COMPILE__)
  extern __shared__ __attribute__((aligned(16))) unsigned char smem[];
  const int tid = threadIdx.x, lane = tid & 63;
  const int wid_all = __builtin_amdgcn_readfirstlane(tid >> 6);
  const int wid = wid_all & 3;
  const int b = blockIdx.x, xcd = b & 7, idx = b >> 3;
  int rank = p.rank;
  unsigned char* myws = p.myws;
  const unsigned char* px = p.x; const unsigned char* pw_ = p.w;
  int wcls = p.wcls;
  __builtin_amdgcn_sched_barrier(0);
  const bool roleS = xcd == 0, roleY = !roleS && (unsigned)(idx - 4) < 24u, roleL = !roleS && !roleY;
  const bool is_loader = (wid_all < 4) != roleL;
  const int yi = idx - 4;
  const int ti = roleS ? idx >> 2 : (roleY ? yi >> 2 : idx & 1);
  int rot = (roleS ? 4 * ti : (int)(0x1B15100B0500ull >> (8 * ti))) + 4 * xcd;
  int voffA[2], voffB[4];
#pragma unroll
  for (int i = 0; i < 2; ++i) { int r = (wid * 2 + i) * 8 + (lane >> 3); voffA[i] = (ti * TM + r) * KD + (((lane & 7) ^ ((r >> 1) & 7)) << 4) - i * 1024; }
#pragma unroll
  for (int i = 0; i < 4; ++i) { int kr = (wid * 4 + i) * 8 + (lane >> 3); voffB[i] = kr * ND + (((lane & 7) ^ (((kr >> 1) & 3) << 1)) << 4) - i * 1024; }
  asm volatile("" : "+v"(voffA[0]), "+v"(voffA[1]), "+v"(voffB[0]), "+v"(voffB[1]), "+v"(voffB[2]), "+v"(voffB[3]));
  asm volatile("" : "+s"(rot));
  asm volatile("" : "+s"(rank), "+s"(myws), "+s"(px), "+s"(pw_), "+s"(wcls));
  gu32* cntp = (gu32*)(myws + WS_CNT + b * 128);
  const unsigned peer = (unsigned)(xcd + rank) & (NR - 1);
  const unsigned srcL = (unsigned)(rank - xcd) & (NR - 1);
  const int tj = (roleL ? (idx >> 1) - wcls : idx) & 3;
  const int n0 = (roleY ? peer : rank) * NSH + tj * TN;
  const unsigned lds0 = (unsigned)(size_t)(LDS_AS unsigned char*)smem;
  gu32* tmo = (gu32*)(myws + WS_TMO);
  unsigned call = 0;

  auto give_up = [&](unsigned long long t0) -> bool {
    const unsigned long long dt = __builtin_amdgcn_s_memrealtime() - t0;
    if (dt < 2000) return false;
    if (dt > DEADLINE_TICKS) return true;
    __builtin_amdgcn_s_sleep(64);
    unsigned tv = 0;
    if (lane == 0) tv = __hip_atomic_load(tmo, __ATOMIC_RELAXED, __HIP_MEMORY_SCOPE_AGENT);
    return __builtin_amdgcn_readfirstlane(tv) == call;
  };
  gu32* xfl = (gu32*)(myws + WS_XFLAG) + ((srcL * 16 + ti * 8 + wid * 2 + (lane & 1)) * 4 + tj) * 32;
  auto wait_xflags = [&](unsigned nblk, int slot) {
    const unsigned long long t0 = __builtin_amdgcn_s_memrealtime();
    const unsigned want = 8 * (call - 1) + nblk;
    unsigned fv = want;
    for (;;) {
      asm volatile("s_mov_b64 exec, 3\n\tglobal_load_dword %0, %1, off sc0 sc1\n\ts_mov_b64 exec, -1\n\ts_waitcnt vmcnt(0)"
                   : "+v"(fv) : "v"(xfl) : "memory");
      if (__builtin_amdgcn_ballot_w64(fv < want) == 0) break;
      if (give_up(t0)) {
        __hip_atomic_store(tmo, call, __ATOMIC_RELAXED, __HIP_MEMORY_SCOPE_AGENT);
        WAITV(0);
        *(LDS_AS unsigned*)(size_t)(lds0 + LDS_DEAD + slot * 4) = 1;
        break;
      }
      __builtin_amdgcn_s_sleep(8);
    }
    __builtin_amdgcn_s_waitcnt(0xC07F);
  };

  if (__builtin_expect(roleL && !is_loader, 0)) {
    if (idx < 4 && peer < (unsigned)NR) {
      const int xo = (idx * 32 + wid * 8 + (lane >> 5)) * KD + (lane & 31) * 16;
      const unsigned char* sp = px + (size_t)YROWS * KD + xo;
      v4u d[32];
      unsigned cntv;
      asm volatile("global_load_dword %0, %1, off sc1" : "=v"(cntv) : "v"(cntp) : "memory");
      x_ldblk<0>(d, sp); x_ldblk<1>(d, sp);
      unsigned char* pw = p.ws[peer & (NR - 1)];
      asm volatile("s_waitcnt vmcnt(8)\n\tv_readfirstlane_b32 %0, %1" : "=s"(call) : "v"(cntv) : "memory");
      call += 1;
      unsigned char* dp = pw + WS_XS + ((call & 1) * NR + rank) * XS_SRC + xo;
      gu32* pf = (gu32*)(pw + WS_XFLAG) + ((rank * 16 + idx * 4 + wid) * 4 + (lane & 3)) * 32;
      const unsigned fbase = 8 * (call - 1);
      static_for<32>([&](auto ic) {
        constexpr int i = decltype(ic)::value;
        constexpr XSched xs = xsched(i);
        vm_wait<xs.wait>(d[i]);
        if (xs.flag && lane < 4) __hip_atomic_store(pf, fbase + xs.flag, __ATOMIC_RELAXED, __HIP_MEMORY_SCOPE_SYSTEM);
        st16(dp + (i >> 2) * 512 + (i & 3) * 2 * KD, d[i]);
        if constexpr ((i & 3) == 0 && i / 4 + 2 < 8) x_ldblk<i / 4 + 2>(d, sp);
      });
      WAITV(0);
      if (lane < 4) __hip_atomic_store(pf, fbase + 8, __ATOMIC_RELAXED, __HIP_MEMORY_SCOPE_SYSTEM);
    }
  }

  if (__builtin_expect(is_loader, 1)) {
    v4i rx = make_rsrc(px, MS_ * KD);
    const v4i rw = make_rsrc(pw_ + n0, KD * ND - n0);
    auto issue = [&](const bool L, int kt, unsigned slot) __attribute__((always_inline)) {
      const int soA = kt * BK, soB = kt * (BK * ND);
      if (L)
        asm volatile("s_mov_b32 m0, %0\n\ts_nop 0\n\tbuffer_load_dwordx4 %1, %3, %4 offen sc0 sc1 lds\n\tbuffer_load_dwordx4 %2, %3, %4 offen offset:1024 sc0 sc1 lds"
                     :: "s"(slot + wid * 2048), "v"(voffA[0]), "v"(voffA[1]), "s"(rx), "s"(soA) : "memory");
      else
        asm volatile("s_mov_b32 m0, %0\n\ts_nop 0\n\tbuffer_load_dwordx4 %1, %3, %4 offen lds\n\tbuffer_load_dwordx4 %2, %3, %4 offen offset:1024 lds"
                     :: "s"(slot + wid * 2048), "v"(voffA[0]), "v"(voffA[1]), "s"(rx), "s"(soA) : "memory");
      asm volatile("s_mov_b32 m0, %0\n\ts_nop 0\n\t"
                   "buffer_load_dwordx4 %1, %5, %6 offen lds\n\t"
                   "buffer_load_dwordx4 %2, %5, %6 offen offset:1024 lds\n\t"
                   "buffer_load_dwordx4 %3, %5, %6 offen offset:2048 lds\n\t"
                   "buffer_load_dwordx4 %4, %5, %6 offen offset:3072 lds"
                   :: "s"(slot + A_BYTES + wid * 4096), "v"(voffB[0]), "v"(voffB[1]), "v"(voffB[2]), "v"(voffB[3]), "s"(rw), "s"(soB) : "memory");
    };
    auto ring = [&](const bool L) __attribute__((always_inline)) {
      auto kstage = [&](int g) { return L ? (g < 16 ? g ^ (ti << 3) : g) : ((g + rot) & (NKT - 1)); };
      static_assert(NST == 5, "prologue");
      issue(L, kstage(0), lds0); issue(L, kstage(1), lds0 + ST_BYTES);
      WAITV(PPL);
      __builtin_amdgcn_s_barrier();
      issue(L, kstage(2), lds0 + 2 * ST_BYTES); issue(L, kstage(3), lds0 + 3 * ST_BYTES);
      WAITV(2 * PPL);
      __builtin_amdgcn_s_barrier();
      issue(L, kstage(4), lds0 + 4 * ST_BYTES); issue(L, kstage(5), lds0);
      unsigned slot = lds0 + ST_BYTES;
      auto run = [&](int g0, int g1) __attribute__((always_inline)) {
#pragma unroll 1
        for (int g = g0; g < g1; ++g) {
          const int kt = kstage(g);
          static_assert((NST - 2) * PPL == 18, "");
#define RING_STEP(POL) \
          asm volatile("s_mov_b32 m0, %0\n\ts_waitcnt vmcnt(18)\n\ts_barrier\n\t" \
                       "buffer_load_dwordx4 %2, %9, %11 offen lds\n\t" \
                       "buffer_load_dwordx4 %3, %9, %11 offen offset:1024 lds\n\t" \
                       "buffer_load_dwordx4 %4, %9, %11 offen offset:2048 lds\n\t" \
                       "buffer_load_dwordx4 %5, %9, %11 offen offset:3072 lds\n\t" \
                       "s_mov_b32 m0, %1\n\ts_nop 0\n\t" \
                       "buffer_load_dwordx4 %6, %8, %10 offen " POL " lds\n\t" \
                       "buffer_load_dwordx4 %7, %8, %10 offen offset:1024 " POL " lds" \
                       :: "s"(slot + A_BYTES + wid * 4096), "s"(slot + wid * 2048), "v"(voffB[0]), "v"(voffB[1]), "v"(voffB[2]), "v"(voffB[3]), \
                          "v"(voffA[0]), "v"(voffA[1]), "s"(rx), "s"(rw), "s"(kt * BK), "s"(kt * (BK * ND)) : "memory")
          if (L) RING_STEP("sc0 sc1"); else RING_STEP("");
#undef RING_STEP
          slot = slot + ST_BYTES == lds0 + NST * ST_BYTES ? lds0 : slot + ST_BYTES;
        }
      };
      if (!L) { run(NST + 1, NKT); return; }
      int g = NST + 1;
#pragma unroll 1
      for (int lv = 0; lv < 2; ++lv) {
        const int gc = lv ? XMID * 4 : XEARLY * 4;
        const unsigned nb = lv ? 8 : XMID;
        run(g, gc - (NST - 2));
        asm volatile("s_mov_b32 m0, %0\n\ts_mov_b64 exec, 3\n\tglobal_load_lds_dword %1, off sc0 sc1\n\ts_mov_b64 exec, -1"
                     :: "s"(lds0 + LDS_XFL + wid * 8), "v"(xfl) : "memory");
        run(gc - (NST - 2), gc);
        WAITV((NST - 2) * PPL);
        const unsigned fv = *(const LDS_AS unsigned*)(size_t)(lds0 + LDS_XFL + wid * 8 + (lane & 1) * 4);
        if (__builtin_amdgcn_ballot_w64(fv < 8 * (call - 1) + nb) != 0) wait_xflags(nb, wid);
        g = gc;
      }
      run(g, NKT);
    };
    if (__builtin_expect(roleL, 0)) {
      call = __builtin_amdgcn_readfirstlane(__hip_atomic_load(cntp, __ATOMIC_RELAXED, __HIP_MEMORY_SCOPE_AGENT)) + 1;
      if (wid == 0) *(LDS_AS unsigned*)(size_t)(lds0 + LDS_CALL) = call - 1;
      rx = make_rsrc(myws + WS_XS + ((call & 1) * NR + srcL) * XS_SRC, XROWS * KD);
      *(LDS_AS unsigned*)(size_t)(lds0 + LDS_DEAD + wid * 4) = 0;
      wait_xflags(XEARLY, wid);
      ring(true);
    } else {
      if (wid == 0)
        asm volatile("s_mov_b32 m0, %0\n\ts_mov_b64 exec, 1\n\tglobal_load_lds_dword %1, off sc1\n\ts_mov_b64 exec, -1"
                     :: "s"(lds0 + LDS_CALL), "v"(cntp) : "memory");
      ring(false);
    }
    static_for<NST - 1>([&](auto ic) {
      constexpr int i = decltype(ic)::value;
      WAITV((NST - 2 - i) * PPL);
      __builtin_amdgcn_s_barrier();
    });
    __builtin_amdgcn_s_barrier();
    asm volatile("" ::: "memory");

    if (roleY) {
      {
        unsigned tq = 0;
#pragma unroll 1
        for (int n = yi * 4 + wid; n < 256; n += 96) {
          const unsigned char* a = pw_ + (size_t)((n >> 1) * 32 + (lane >> 1)) * ND + rank * NSH + (((n & 1) - wcls) & 3) * 128 + (lane & 1) * 64;
          asm volatile("global_load_dword %0, %1, off" : "+v"(tq) : "v"(a) : "memory");
        }
        asm volatile("s_waitcnt vmcnt(0)" : "+v"(tq) :: "memory");
      }
      call = __builtin_amdgcn_readfirstlane(*(const LDS_AS unsigned*)(size_t)(lds0 + LDS_CALL)) + 1;
      const int id = ((xcd - 1) * 24 + yi) * 4 + wid;
      const int si = id / 96, c96 = id - si * 96;
      const unsigned src = (unsigned)(rank + 1 + si) & (NR - 1);
      const int cbk = c96 & 15;
      unsigned* rb = (unsigned*)(smem + 8 * CBUF_BYTES + wid * RBUF_BYTES);
      unsigned* tab = rb + CHUNK_BYTES / 4;
      const unsigned rb_lds = lds0 + 8 * CBUF_BYTES + wid * RBUF_BYTES;
      const unsigned long long t0 = __builtin_amdgcn_s_memrealtime();
      bool dead = false;
#pragma unroll 1
      for (int blk = 0; blk < 2; ++blk) {
        const int rb32 = (c96 >> 4) * 2 + blk;
        unsigned char* sp0 = myws + WS_SLOT + ((call & 1) * NR + src) * SLOT_SRC + (size_t)(rb32 * 16 + cbk) * CHUNK_BYTES;
        v4u hd = {~0u, ~0u, ~0u, ~0u};
        for (;;) {
          asm volatile("s_mov_b64 exec, 0xff\n\tglobal_load_dwordx4 %0, %1, off sc0 sc1\n\ts_mov_b64 exec, -1\n\ts_waitcnt vmcnt(0)" : "+v"(hd) : "v"(sp0 + lane * 16) : "memory");
          const unsigned lo = min(min(hd[0] & 0xffffu, hd[1] & 0xffffu), min(hd[2] & 0xffffu, hd[3] & 0xffffu));
          const unsigned hi = min(min(hd[0], hd[1]), min(hd[2], hd[3]));
          if (__builtin_amdgcn_ballot_w64(lo == 0 || hi < 0x10000u) == 0) break;
          if (dead || give_up(t0)) { dead = true; break; }
          __builtin_amdgcn_s_sleep(4);
        }
        if (lane < 8) *(v4u*)(rb + lane * 4) = hd;
        const unsigned msk = ((const unsigned short*)rb)[lane];
        const unsigned pc = __builtin_popcount(msk);
        const unsigned inc = wave_scan(pc);
        const unsigned nby = (128 + 4 * __builtin_amdgcn_readlane(inc, 63) + 15) & ~15u;
        const v4i rs = make_rsrc(sp0, nby);
        v4u d[4];
        unsigned of[4];
#pragma unroll
        for (int i = 0; i < 4; ++i) { of[i] = 128 + i * 1024u + lane * 16u; d[i] = (v4u){0, 0, 0, 0}; }
        unsigned pend = (nby > 128 + 3072) ? 15u : (nby > 128 + 2048) ? 7u : (nby > 128 + 1024) ? 3u : 1u;
        auto fetch = [&]() {
#pragma unroll
          for (int i = 0; i < 4; ++i)
            if (pend >> i & 1) asm volatile("s_nop 4\n\tbuffer_load_dwordx4 %0, %1, %2, 0 offen sc0 sc1" : "+v"(d[i]) : "v"(of[i]), "s"(rs) : "memory");
        };
        fetch();
        tab[lane] = ((32 + inc - pc) << 16) | msk;
        unsigned ga[4][4], gm[4][4];
#pragma unroll
        for (int i = 0; i < 4; ++i) {
          const unsigned t = tab[i * 8 + (lane >> 3) + 32 * (lane & 1)];
          const unsigned sh = (lane & 6) * 2;
          const unsigned a = rb_lds + 4 * ((t >> 16) + __builtin_popcount(t & ((1u << sh) - 1u)));
          const unsigned nib = t >> sh;
          ga[i][0] = a;
          asm("v_bfe_i32 %0, %7, 0, 1\n\tv_bfe_i32 %1, %7, 1, 1\n\tv_bfe_i32 %2, %7, 2, 1\n\tv_bfe_i32 %3, %7, 3, 1\n\t"
              "v_mad_i32_i24 %4, %0, -4, %8\n\tv_mad_i32_i24 %5, %1, -4, %4\n\tv_mad_i32_i24 %6, %2, -4, %5"
              : "=&v"(gm[i][0]), "=&v"(gm[i][1]), "=&v"(gm[i][2]), "=&v"(gm[i][3]), "=&v"(ga[i][1]), "=&v"(ga[i][2]), "=&v"(ga[i][3]) : "v"(nib), "v"(a));
        }
        for (;;) {
          asm volatile("s_waitcnt vmcnt(0)" : "+v"(d[0]), "+v"(d[1]), "+v"(d[2]), "+v"(d[3]) :: "memory");
#pragma unroll
          for (int i = 0; i < 4; ++i)
            if (pend >> i & 1) {
              const bool miss = of[i] < nby && min(min(d[i][0], d[i][1]), min(d[i][2], d[i][3])) == 0;
              if (__builtin_amdgcn_ballot_w64(miss) == 0) { pend &= ~(1u << i); *(v4u*)((unsigned char*)rb + of[i]) = d[i]; }
            }
          if (pend == 0 || dead) break;
          if (give_up(t0)) { dead = true; break; }
          fetch();
        }
        float* dst = p.out + (size_t)(src * MS_ + rb32 * 32 + (lane >> 3)) * NSH + cbk * 32 + (lane & 7) * 4;
        const unsigned nanv = dead ? 0x7fc00000u : 0u;
        v4u o[4];
#pragma unroll
        for (int i = 0; i < 4; ++i)
#pragma unroll
          for (int j = 0; j < 4; ++j) o[i][j] = *(const LDS_AS unsigned*)(size_t)ga[i][j];
#pragma unroll
        for (int i = 0; i < 4; ++i) {
#pragma unroll
          for (int j = 0; j < 4; ++j) o[i][j] = (o[i][j] & gm[i][j]) | nanv;
          st16(dst + i * 8 * NSH, o[i]);
        }
        const v4u zz = {0, 0, 0, 0};
        const unsigned lo16 = lane * 16;
        asm volatile("buffer_store_dwordx4 %0, %1, %2, 0 offen sc0 sc1\n\tbuffer_store_dwordx4 %0, %1, %2, 0 offen offset:1024 sc0 sc1\n\t"
                     "buffer_store_dwordx4 %0, %1, %2, 0 offen offset:2048 sc0 sc1\n\ts_nop 1" :: "v"(zz), "v"(lo16), "s"(rs) : "memory");
        if (nby > 3072)
          asm volatile("buffer_store_dwordx4 %0, %1, %2, 0 offen offset:3072 sc0 sc1\n\tbuffer_store_dwordx4 %0, %3, %2, 0 offen sc0 sc1\n\ts_nop 1"
                       :: "v"(zz), "v"(lo16), "s"(rs), "v"(lo16 + 4096) : "memory");
      }
      if (dead) __hip_atomic_store(tmo, call, __ATOMIC_RELAXED, __HIP_MEMORY_SCOPE_AGENT);
    }
  } else {
    float sxv = *(const CONST_AS float*)p.sx, swv = *(const CONST_AS float*)p.sw;
    unsigned char* pws = p.ws[peer & (NR - 1)];
    float* pout = p.out;
    asm volatile("" : "+s"(sxv), "+s"(swv), "+s"(pws), "+s"(pout));
    const float scale = sxv * swv;
    const int r = lane & 31, h = lane >> 5;
    const int aoff = r * 128 + (((2 * h) ^ ((r >> 1) & 7)) << 4);
    const int bkr = 32 * h + ((lane & 15) >> 1);
    const int boff = A_BYTES + bkr * TN + (((wid * 2 + ((lane >> 4) & 1)) ^ (((bkr >> 1) & 3) << 1)) << 4) + 8 * (lane & 1);
    struct Frag { v8i a[2][2], b[2]; };
    auto ldfrag = [&](Frag& f, int slotidx) {
      const unsigned char* slot = smem + slotidx * ST_BYTES;
#pragma unroll
      for (int kk = 0; kk < 2; ++kk) {
        const unsigned char* bp = slot + boff + kk * 64 * TN;
        v2i b0 = __builtin_amdgcn_ds_read_tr8_b64_v2i32((LDS_AS v2i*)(bp));
        v2i b1 = __builtin_amdgcn_ds_read_tr8_b64_v2i32((LDS_AS v2i*)(bp + 8 * TN));
        v2i b2 = __builtin_amdgcn_ds_read_tr8_b64_v2i32((LDS_AS v2i*)(bp + 16 * TN));
        v2i b3 = __builtin_amdgcn_ds_read_tr8_b64_v2i32((LDS_AS v2i*)(bp + 24 * TN));
        v4i l = __builtin_shufflevector(b0, b1, 0, 1, 2, 3), hh = __builtin_shufflevector(b2, b3, 0, 1, 2, 3);
        f.b[kk] = __builtin_shufflevector(l, hh, 0, 1, 2, 3, 4, 5, 6, 7);
        const int o = aoff ^ (kk << 6);
#pragma unroll
        for (int i = 0; i < 2; ++i) {
          v4i lo = *(const v4i*)(slot + o + i * 4096), hi = *(const v4i*)(slot + (o ^ 16) + i * 4096);
          f.a[kk][i] = __builtin_shufflevector(lo, hi, 0, 1, 2, 3, 4, 5, 6, 7);
        }
      }
    };
    v16f acc[2] = {};
    Frag f0, f1;
    int sl = 0;
    auto step = [&](Frag& cur, Frag& nxt) {
      __builtin_amdgcn_s_waitcnt(0xC07F);
      __builtin_amdgcn_sched_barrier(0);
      __builtin_amdgcn_s_barrier();
      asm volatile("" ::: "memory");
      __builtin_amdgcn_sched_barrier(0);
      sl = sl + 1 == NST ? 0 : sl + 1;
      ldfrag(nxt, sl);
#pragma unroll
      for (int kk = 0; kk < 2; ++kk)
#pragma unroll
        for (int i = 0; i < 2; ++i)
          acc[i] = __builtin_amdgcn_mfma_scale_f32_32x32x64_f8f6f4(cur.b[kk], cur.a[kk][i], acc[i], 1, 0, 0, 0, 0, 0);
#pragma unroll
      for (int i = 0; i < 4; ++i) { __builtin_amdgcn_sched_group_barrier(0x8, 1, 0); __builtin_amdgcn_sched_group_barrier(0x100, 4, 0); }
      __builtin_amdgcn_sched_barrier(0);
    };
    __builtin_amdgcn_s_barrier();
    asm volatile("" ::: "memory");
    call = __builtin_amdgcn_readfirstlane(*(const LDS_AS unsigned*)(size_t)(lds0 + LDS_CALL)) + 1;
    asm volatile("" : "+s"(call));
    ldfrag(f0, 0);
#pragma unroll 1
    for (int g = 0; g < NKT - 2; g += 2) { step(f0, f1); step(f1, f0); }
    step(f0, f1);
#pragma unroll
    for (int i = 0; i < 2; ++i)
#pragma unroll
      for (int kk = 0; kk < 2; ++kk)
        acc[i] = __builtin_amdgcn_mfma_scale_f32_32x32x64_f8f6f4(f1.b[kk], f1.a[kk][i], acc[i], 1, 0, 0, 0, 0, 0);
    __builtin_amdgcn_sched_barrier(0);
    __builtin_amdgcn_s_barrier();
    asm volatile("" ::: "memory");
    if (!roleY) {
      const bool bad = roleL && __builtin_amdgcn_ballot_w64(*(const LDS_AS unsigned*)(size_t)(lds0 + LDS_DEAD + (lane & 3) * 4) != 0) != 0;
      const int orow = roleL ? srcL * MS_ + YROWS + ti * TM : rank * MS_ + ti * TM;
      unsigned char* cb = smem + wid * 2 * CBUF_BYTES;
      const float sc2 = bad ? __builtin_nanf("") : scale;
      float* dst = pout + (size_t)(orow + (lane >> 3)) * NSH + tj * TN + wid * 32 + (lane & 7) * 4;
#pragma unroll
      for (int i = 0; i < 2; ++i) {
#pragma unroll
        for (int q = 0; q < 4; ++q) {
          v4u o;
#pragma unroll
          for (int j = 0; j < 4; ++j) o[j] = __float_as_uint(__builtin_elementwise_maximum(acc[i][q * 4 + j] * sc2, 0.f));
          *(v4u*)(cb + (i * 32 + r) * 128 + (((q * 2 + h) ^ (r & 7)) << 4)) = o;
        }
        v4u o[4];
#pragma unroll
        for (int q = 0; q < 4; ++q) o[q] = *(const v4u*)(cb + (i * 4 + q) * 1024 + (lane >> 3) * 128 + (((lane & 7) ^ (lane >> 3)) << 4));
#pragma unroll
        for (int q = 0; q < 4; ++q) st16(dst + (i * 4 + q) * 8 * NSH, o[q]);
      }
    } else {
      unsigned char* dbase = pws + WS_SLOT + ((call & 1) * NR + rank) * SLOT_SRC + (size_t)(ti * 32 + tj * 4 + wid) * CHUNK_BYTES;
      unsigned nbys[2], off = 0;
      auto piece = [&](int blk, unsigned nby) {
        const unsigned end = nby - off <= YPIECE + YTAIL ? nby : off + YPIECE;
        const v4i rs = make_rsrc(dbase + blk * 16 * CHUNK_BYTES, end);
        const unsigned o = off + lane * 16;
        v4u v = *(const v4u*)(smem + (wid * 2 + blk) * CBUF_BYTES + o);
        asm volatile("s_nop 4\n\tbuffer_store_dwordx4 %0, %1, %2, 0 offen sc0 sc1\n\ts_nop 1" :: "v"(v), "v"(o), "s"(rs) : "memory");
        off = end;
      };
#pragma unroll
      for (int blk = 0; blk < 2; ++blk) {
        unsigned* cb = (unsigned*)(smem + (wid * 2 + blk) * CBUF_BYTES);
        const unsigned cb_lds = lds0 + (wid * 2 + blk) * CBUF_BYTES;
        float u[16];
        unsigned msk = 0;
#pragma unroll
        for (int e = 0; e < 16; ++e) u[e] = acc[blk][e] * scale;
#define CA(n) "v_cmp_nge_f32 vcc, 0, %" #n "\n\tv_addc_co_u32 %0, vcc, %0, %0, vcc\n\t"
        asm volatile(CA(16) CA(15) CA(14) CA(13) CA(12) CA(11) CA(10) CA(9) CA(8) CA(7) CA(6) CA(5) CA(4) CA(3) CA(2) CA(1)
            : "+v"(msk) : "v"(u[0]), "v"(u[1]), "v"(u[2]), "v"(u[3]), "v"(u[4]), "v"(u[5]), "v"(u[6]), "v"(u[7]),
              "v"(u[8]), "v"(u[9]), "v"(u[10]), "v"(u[11]), "v"(u[12]), "v"(u[13]), "v"(u[14]), "v"(u[15]) : "vcc");
#undef CA
        if (msk == 0) u[0] = -0.0f;
        msk = max(msk, 1u);
        const unsigned pc = __builtin_popcount(msk);
        const unsigned inc = wave_scan(pc);
        *(LDS_AS unsigned short*)(size_t)(cb_lds + lane * 2) = (unsigned short)msk;
        unsigned a = cb_lds + 4 * (32 + inc - pc), t0_, t1_;
        const unsigned dump = cb_lds + CHUNK_BYTES + 16 + r * 4;
#define PKE(e, n) "v_bfe_i32 %1, %3, " #e ", 1\n\tv_bfi_b32 %2, %1, %0, %4\n\tv_mad_i32_i24 %0, %1, -4, %0\n\tds_write_b32 %2, %" #n "\n\t"
        asm volatile(PKE(0, 5) PKE(1, 6) PKE(2, 7) PKE(3, 8) PKE(4, 9) PKE(5, 10) PKE(6, 11) PKE(7, 12)
                     PKE(8, 13) PKE(9, 14) PKE(10, 15) PKE(11, 16) PKE(12, 17) PKE(13, 18) PKE(14, 19) PKE(15, 20)
            : "+v"(a), "=&v"(t0_), "=&v"(t1_) : "v"(msk), "v"(dump), "v"(u[0]), "v"(u[1]), "v"(u[2]), "v"(u[3]), "v"(u[4]), "v"(u[5]), "v"(u[6]), "v"(u[7]),
              "v"(u[8]), "v"(u[9]), "v"(u[10]), "v"(u[11]), "v"(u[12]), "v"(u[13]), "v"(u[14]), "v"(u[15]) : "memory");
#undef PKE
        const unsigned cnt = 32 + __builtin_amdgcn_readlane(inc, 63);
        if (lane < 3) cb[cnt + lane] = 1u;
        nbys[blk] = (cnt * 4 + 15) & ~15u;
        if (blk == 0) {
          piece(0, nbys[0]);
          __builtin_amdgcn_sched_barrier(0);
        }
      }
#pragma unroll 1
      for (int blk = 0; blk < 2; ++blk) {
        const unsigned nby = blk ? nbys[1] : nbys[0];
        while (off < nby) { WAITV(0); piece(blk, nby); }
        off = 0;
      }
    }
    if (wid == 0 && lane == 0) __hip_atomic_store(cntp, call, __ATOMIC_RELAXED, __HIP_MEMORY_SCOPE_AGENT);
  }
#endif
}

extern "C" void kernel_launch(int rank, int nranks, ncclComm_t comm, hipStream_t stream,
                              void* const* d_in, const int* in_sizes, int n_in,
                              void* d_out, int out_size, const int* in_dtypes, int out_dtype,
                              void* const* d_out_all, void* const* d_ws_all, size_t ws_bytes) {
  if (nranks != NR || ws_bytes < WS_NEED || (unsigned)rank >= (unsigned)NR || n_in < 4 ||
      in_sizes[0] != MS_ * KD || in_sizes[1] != KD * ND || out_size != NR * MS_ * NSH) return;
  static bool init = false;
  if (!init) {
    HIP_CHECK(hipFuncSetAttribute((const void*)fused_gemm_a2a, hipFuncAttributeMaxDynamicSharedMemorySize, LDS_BYTES));
    init = true;
  }
  Params p;
  p.x = (const unsigned char*)d_in[0]; p.w = (const unsigned char*)d_in[1];
  p.sx = (const float*)d_in[2]; p.sw = (const float*)d_in[3];
  p.out = (float*)d_out;
  for (int i = 0; i < NR; ++i) p.ws[i] = (unsigned char*)d_ws_all[i];
  p.myws = p.ws[rank];
  p.rank = rank; p.wcls = (int)(((size_t)p.w >> 7) & 3);
  fused_gemm_a2a<<<GRID, 512, LDS_BYTES, stream>>>(p);
  HIP_CHECK(hipGetLastError());
}
```
